```python
import numpy as np

kernel_name = "gpu_gemm_rs_m4096_k4096_n4096_bf16_gelu"


ABSMAX_RTOL = 0.008

COMPUTE_DTYPE = "bfloat16"
EPILOGUE = "gelu"
IN_QUANT = None
OUT_QUANT = None
IN_DTYPES = {"x": "bfloat16", "w_mat": "bfloat16"}
WORLD_SIZE = 8

def _to_bf16(x):
    u = np.ascontiguousarray(x, dtype=np.float32).view(np.uint32)
    return ((u + 0x7FFF + ((u >> 16) & 1)) & 0xFFFF0000).view(np.float32)

def setup_inputs():

    x = _to_bf16(np.random.randn(4096, 4096).astype(np.float32))
    w_mat = _to_bf16((np.random.randn(4096, 4096) / np.sqrt(4096)).astype(np.float32))
    return {"x": x, "w_mat": w_mat}

def _epilogue(y):
    c = 0.7978845608028654
    return (0.5 * y * (1.0 + np.tanh(c * (y + 0.044715 * y ** 3)))).astype(np.float32)

def reference(x, w_mat):
    y = (x @ w_mat).astype(np.float32)
    return _epilogue(y)
```

```cpp
#include <hip/hip_runtime.h>
#include <rccl/rccl.h>
#include <cstdio>
#include <stdint.h>

#define HIP_CHECK(expr) do { hipError_t _e = (expr); if (_e != hipSuccess) { \
    fprintf(stderr, "HIP error %s at %s:%d\n", hipGetErrorString(_e), __FILE__, __LINE__); \
    return; } } while (0)

typedef short bf16x8 __attribute__((ext_vector_type(8)));
typedef float f32x16 __attribute__((ext_vector_type(16)));
typedef unsigned u32x4 __attribute__((ext_vector_type(4)));
typedef __bf16 bf16x2 __attribute__((ext_vector_type(2)));
typedef float f32x2 __attribute__((ext_vector_type(2)));
#define DI __device__ __forceinline__

constexpr int NR = 8;
constexpr unsigned MiB = 1u << 20;
constexpr unsigned OFF_XS = 0;
constexpr unsigned OFF_WS = 8 * MiB;
constexpr unsigned OFF_PS = 18 * MiB;
constexpr unsigned PAR = 24 * MiB;
constexpr unsigned OFF_LP = 2 * PAR;
constexpr unsigned OFF_CTL = OFF_LP + 8 * MiB;
constexpr unsigned L_CX = 0;
constexpr unsigned L_CW = L_CX + 128;
constexpr unsigned L_FP = L_CW + 160;
constexpr unsigned L_SEQ = L_FP + 192;
constexpr unsigned L_TMO = L_SEQ + 256;
constexpr unsigned L_QT = L_TMO + 1;
constexpr unsigned L_Q = L_QT + 8;
constexpr unsigned L_END = L_Q + 16;
constexpr unsigned OFF_DBG = OFF_CTL + 512 * 1024;
constexpr unsigned WS_NEED = OFF_CTL + MiB;
static_assert(L_END * 128 <= 512 * 1024, "ctl");
#define NWX 2
constexpr unsigned long long DEADLINE = 100000000ull;
constexpr int AUX_SYS = 17;

constexpr int RING = 4, STAGE_B = 32768, LDS_POLL = RING * STAGE_B, LDS_BYTES = LDS_POLL + 2048;

struct Params {
    const uint16_t* x; const uint16_t* w; uint16_t* out;
    char* ws[NR];
    int rank;
};

#define TS(i) do {} while (0)
#define TSW(i) do {} while (0)

DI unsigned* ctl(char* ws, unsigned line) { return (unsigned*)(ws + OFF_CTL + line * 128u); }
DI __amdgpu_buffer_rsrc_t rsrc(const void* p, unsigned bytes) { return __builtin_amdgcn_make_buffer_rsrc((void*)p, 0, bytes, 0x00020000); }
DI int rfl(int v) { return __builtin_amdgcn_readfirstlane(v); }
#define WAIT_V(n) asm volatile("s_waitcnt vmcnt(" #n ")" ::: "memory")
#define BAR() do { asm volatile("s_waitcnt lgkmcnt(0)" ::: "memory"); __builtin_amdgcn_s_barrier(); asm volatile("" ::: "memory"); } while (0)

#define LDFL " sc0 sc1"
DI void glds4k(const char* g, unsigned lds) {
    unsigned keep;
    asm volatile("s_mov_b32 %0, m0\n\ts_mov_b32 m0, %2\n\ts_nop 0\n\t"
                 "global_load_lds_dwordx4 %1, off" LDFL "\n\t"
                 "global_load_lds_dwordx4 %1, off offset:1024" LDFL "\n\t"
                 "global_load_lds_dwordx4 %1, off offset:2048" LDFL "\n\t"
                 "global_load_lds_dwordx4 %1, off offset:3072" LDFL "\n\t"
                 "s_mov_b32 m0, %0" : "=&s"(keep) : "v"(g), "s"(lds) : "memory");
}
template <int OFS> DI void glds1k(const char* g, unsigned lds) {
    unsigned keep;
    asm volatile("s_mov_b32 %0, m0\n\ts_mov_b32 m0, %2\n\ts_nop 0\n\t"
                 "global_load_lds_dwordx4 %1, off offset:%c3" LDFL "\n\t"
                 "s_mov_b32 m0, %0" : "=&s"(keep) : "v"(g), "s"(lds), "i"(OFS) : "memory");
}
DI void glds_word(const unsigned* g, unsigned lds) {
    unsigned keep;
    asm volatile("s_mov_b32 %0, m0\n\ts_mov_b32 m0, %2\n\ts_nop 0\n\t"
                 "global_load_lds_dword %1, off sc0 sc1\n\t"
                 "s_mov_b32 m0, %0" : "=&s"(keep) : "v"(g), "s"(lds) : "memory");
}

DI unsigned long long wave_wait(const unsigned* p, bool active, unsigned target, unsigned long long need, unsigned* tmo, unsigned call) {
    unsigned long long t0 = __builtin_amdgcn_s_memrealtime();
    for (unsigned it = 0;; ++it) {
        unsigned v = 0;
        if (active) v = __hip_atomic_load(p, __ATOMIC_RELAXED, __HIP_MEMORY_SCOPE_SYSTEM);
        unsigned long long ok = __builtin_amdgcn_ballot_w64(active && v >= target);
        if ((ok & need) == need) return ok;
        if ((it & 63) == 63) {
            unsigned t = rfl(__hip_atomic_load(tmo, __ATOMIC_RELAXED, __HIP_MEMORY_SCOPE_AGENT));
            if (t == call || __builtin_amdgcn_s_memrealtime() - t0 > DEADLINE) {
                __hip_atomic_store(tmo, call, __ATOMIC_RELAXED, __HIP_MEMORY_SCOPE_AGENT);
                return 0;
            }
        }
        __builtin_amdgcn_s_sleep(4);
    }
}

DI int brick_row(int r, int rb) {
    if (rb < 4) return 512 * (r ^ 1) + 128 * rb;
    if (rb < 12) return 512 * (r ^ (4 + (rb & 3))) + 256 * ((r >> 1) & 1) + 128 * ((rb - 4) >> 2);
    return 512 * r + 128 * (rb - 12);
}

#define ORDER_LIST 0, 1, 2, 3, 16, 17, 18, 19, 20, 21, 22, 23, 24, 25, 26, 27, 28, 29, 30, 31, 4, 5, 6, 7, 8, 9, 10, 11, 12, 13, 14, 15
constexpr unsigned char ORDER[32] = {ORDER_LIST};
constexpr unsigned long long ord8(int g) { unsigned long long v = 0; for (int i = 0; i < 8; ++i) v |= (unsigned long long)ORDER[8 * g + i] << (8 * i); return v; }
constexpr unsigned long long pos8() { unsigned long long v = 0; for (int q = 0; q < 32; ++q) if (ORDER[q] >= 4 && ORDER[q] < 12) v |= (unsigned long long)q << (8 * (ORDER[q] - 4)); return v; }
DI int order_at(int q) {
    const unsigned long long v = (q & 16) ? ((q & 8) ? ord8(3) : ord8(2)) : ((q & 8) ? ord8(1) : ord8(0));
    return (int)(v >> (8 * (q & 7))) & 31;
}
struct Piece { int kind; int seg; int blk; int sub; };
DI Piece decode_local(int n) {
    Piece p;
    int rnd = n >= 768; n -= rnd * 768;
    p.sub = n & 15;
    if (n < 512) { p.kind = 1; p.seg = rnd * 8 + (n >> 8); p.blk = (n >> 4) & 15; }
    else { n -= 512; p.kind = 0; p.seg = n >> 7; int q = (n >> 4) & 7; p.blk = rnd ? 4 + q : (q < 4 ? q : q + 8); }
    return p;
}
DI void piece_load(const Params& P, const Piece& p, int dst, int lane, unsigned (&r)[16]) {
    const int k0 = 256 * ((p.seg >= 2 && p.seg < 8) ? (dst & 1) : (p.seg & 1)) + 64 * (p.sub >> 2);
    if (p.kind == 0) {
        const int row = brick_row(dst, p.blk) + 32 * (p.sub & 3) + (lane & 31);
        const u32x4* s = (const u32x4*)(P.x + (size_t)row * 512 + k0 + 8 * (lane >> 5));
#pragma unroll
        for (int ks = 0; ks < 4; ++ks) { u32x4 v = s[2 * ks]; r[4 * ks] = v[0]; r[4 * ks + 1] = v[1]; r[4 * ks + 2] = v[2]; r[4 * ks + 3] = v[3]; }
    } else {
        const int col = 2048 * ((dst >> 2) ^ (p.seg >> 3)) + 128 * p.blk + 64 * ((p.sub >> 1) & 1) + 2 * (lane & 31);
        const int k = k0 + 32 * (p.sub & 1) + 8 * (lane >> 5);
        const unsigned* s = (const unsigned*)(P.w + (size_t)k * 4096 + col);
#pragma unroll
        for (int ks = 0; ks < 2; ++ks)
#pragma unroll
            for (int j = 0; j < 8; ++j) r[8 * ks + j] = s[(16 * ks + j) * 2048];
    }
}
DI int ps_index(int r, int t) {
    const int rb = t >> 4, cb = t & 15;
    return rb < 4 ? t : (1 + (r & 1)) * 64 + (2 * ((r >> 1) & 1) + ((rb - 4) >> 2)) * 16 + cb;
}
DI void piece_xform(const Piece& p, const unsigned (&r)[16], u32x4 (&o)[4]) {
    if (p.kind == 1) {
#pragma unroll
        for (int ks = 0; ks < 2; ++ks)
#pragma unroll
            for (int m = 0; m < 4; ++m) {
                o[ks][m] = __builtin_amdgcn_perm(r[8 * ks + 2 * m + 1], r[8 * ks + 2 * m], 0x05040100u);
                o[2 + ks][m] = __builtin_amdgcn_perm(r[8 * ks + 2 * m + 1], r[8 * ks + 2 * m], 0x07060302u);
            }
    } else {
#pragma unroll
        for (int ks = 0; ks < 4; ++ks) o[ks] = u32x4{r[4 * ks], r[4 * ks + 1], r[4 * ks + 2], r[4 * ks + 3]};
    }
    asm volatile("" : "+v"(o[0]), "+v"(o[1]), "+v"(o[2]), "+v"(o[3]) :: "memory");
}
DI void piece_store(const Params& P, const Piece& p, int dst, int lane, unsigned par, const u32x4 (&o)[4]) {
    __amdgpu_buffer_rsrc_t rs = rsrc(P.ws[dst & 7], OFF_LP);
    const unsigned off = par + p.sub * 4096u + (p.kind ? OFF_WS : OFF_XS) + p.seg * MiB + p.blk * 65536u;
#pragma unroll
    for (int q = 0; q < 4; ++q) __builtin_amdgcn_raw_buffer_store_b128(o[q], rs, lane * 16 + q * 1024, off, AUX_SYS);
}
DI void piece_flag(const Params& P, const Piece& p, int dst, int lane) {
    const unsigned line = (p.kind ? L_CW : L_CX) + p.seg * 16 + p.blk;
    if (lane == 0) __hip_atomic_fetch_add(ctl(P.ws[dst & 7], line), 1u, __ATOMIC_RELAXED, __HIP_MEMORY_SCOPE_SYSTEM);
}

DI unsigned pack2(float a, float b) { f32x2 v = {a, b}; return __builtin_bit_cast(unsigned, __builtin_convertvector(v, bf16x2)); }
DI float gelu(float y) {
    const float c2 = -2.0f * 0.7978845608028654f * 1.4426950408889634f;
    float u = y + 0.044715f * y * y * y;
    float e = __builtin_amdgcn_exp2f(c2 * u);
    return y * __builtin_amdgcn_rcpf(1.0f + e);
}

DI void poison(f32x16 (&a)[2][2]) {
#pragma unroll
    for (int m = 0; m < 2; ++m)
#pragma unroll
        for (int i = 0; i < 2; ++i)
#pragma unroll
            for (int e = 0; e < 16; ++e) a[m][i][e] = __builtin_nanf("");
}

__global__ __launch_bounds__(256, 1) void fused_gemm_rs(Params P) {
    __shared__ __attribute__((aligned(1024))) char lds[LDS_BYTES];
    const int b = blockIdx.x, lane = threadIdx.x & 63, wave = rfl(threadIdx.x >> 6);
    const int xcd = b & 7, wi = b >> 3;
    const int rank = P.rank;
    char* const me = P.ws[rank & 7];
    TS(0);
    const unsigned seq0 = __hip_atomic_load(ctl(me, L_SEQ + b), __ATOMIC_RELAXED, __HIP_MEMORY_SCOPE_AGENT);
    unsigned* const tmo = ctl(me, L_TMO);
    unsigned raw[16] = {};
    u32x4 o[4];

    if (wi < 8) {
        const int i = wi * 4 + wave, d = 1 + i % 7, dst = rank ^ d;
        const int nin = d == 1 ? 0 : (d < 4 ? 512 : 480);
        const int nend = i >= 7 * NWX ? 0 : nin + (d == 1 ? 512 : (d < 4 ? 0 : 256));
        int sk1 = 99, sk2 = 99;
        if (d >= 4) { const int a = (pos8() >> (8 * (d - 4))) & 31, c = (pos8() >> (8 * d)) & 31; sk1 = a < c ? a : c; sk2 = a < c ? c : a; }
        const int lseg = (d >> 2) * 8 + (dst & 1);
        const unsigned vx = (lane & 31) * 1024 + (lane >> 5) * 16, vl = lane * 16;
        const __amdgpu_buffer_rsrc_t rd = rsrc(P.ws[dst & 7], OFF_LP);
        unsigned call = 0, par = 0;
        auto blk_of = [&](int u) { int q = u >> 4; q += q >= sk1; q += q >= sk2; return order_at(q & 31); };
        auto rdy_ptr = [&](int u) { return u >= nin ? ctl(me, L_Q + 2 * d) + (((u - nin) >> 3) & 63) : ctl(me, L_CW + lseg * 16 + (blk_of(u) & 15)); };
        auto rdy_thr = [&](int u) { return u >= nin ? call << 8 : (blk_of(u) >= 16 ? 16 * call : 0u); };
        u32x4 n[4];
        unsigned ndst, nflag;
        auto fetch = [&](int u, unsigned e) {
            unsigned so, step = 1024, vo = vl, bytes = OFF_CTL;
            const char* base = me;
            if (u >= nin) {
                const int t = (e & 255) < 192 ? (e & 255) : 0, sub = (u - nin) & 7, pi = ps_index(rank, t);
                so = OFF_LP + t * 32768u + sub * 4096u; ndst = OFF_PS + pi * 32768u + sub * 4096u; nflag = L_FP + pi;
            } else {
                const int bk = blk_of(u), blk = bk & 15, sub = u & 15;
                if (bk >= 16) {
                    so = par + OFF_WS + lseg * MiB + blk * 65536u + sub * 4096u;
                    ndst = OFF_WS + d * MiB + blk * 65536u + sub * 4096u; nflag = L_CW + d * 16 + blk;
                } else {
                    base = (const char*)P.x; bytes = 4096 * 1024; step = 32; vo = vx;
                    so = (brick_row(dst, blk) + 32 * (sub & 3)) * 1024 + (dst & 1) * 512 + (sub >> 2) * 128;
                    ndst = OFF_XS + d * MiB + blk * 65536u + sub * 4096u; nflag = L_CX + d * 16 + blk;
                }
            }
            const __amdgpu_buffer_rsrc_t rs = rsrc(base, bytes);
#pragma unroll
            for (int q = 0; q < 4; ++q) n[q] = __builtin_amdgcn_raw_buffer_load_b128(rs, vo, so + q * step, AUX_SYS);
        };
        auto take = [&]() {
#pragma unroll
            for (int q = 0; q < 4; ++q) o[q] = n[q];
            asm volatile("" : "+v"(o[0]), "+v"(o[1]), "+v"(o[2]), "+v"(o[3]) :: "memory");
        };
        auto flag = [&](unsigned line) { if (lane == 0) __hip_atomic_fetch_add(ctl(P.ws[dst & 7], line), 1u, __ATOMIC_RELAXED, __HIP_MEMORY_SCOPE_SYSTEM); };
        int u = (i / 7) * 8 + xcd;
        fetch(u, 0);
        call = rfl(seq0) + 1; par = (call & 1) * PAR;
        take();
        bool ok = true;
        auto slow = [&](int uu) {
            const unsigned* rp = rdy_ptr(uu);
            if (!wave_wait(rp, lane == 0, rdy_thr(uu), 1ull, tmo, call)) ok = false;
            fetch(uu, rfl(__hip_atomic_load(rp, __ATOMIC_RELAXED, __HIP_MEMORY_SCOPE_AGENT)));
            take();
        };
        if (u < nend) {
            if (rdy_thr(u) != 0) slow(u);
            unsigned ent = 0, pflag = ~0u;
            while (ok) {
                const unsigned cdst = ndst, cflag = nflag;
                const int un = u + 8 * NWX, un2 = un + 8 * NWX;
                const bool more = un < nend, early = more && ent >= rdy_thr(un);
                fetch(early ? un : u, ent);
                const unsigned e2 = __hip_atomic_load(rdy_ptr(un2), __ATOMIC_RELAXED, __HIP_MEMORY_SCOPE_AGENT);
#pragma unroll
                for (int q = 0; q < 4; ++q) __builtin_amdgcn_raw_buffer_store_b128(o[q], rd, vl + q * 1024, par + cdst, AUX_SYS);
                WAIT_V(4);
                take();
                ent = rfl(e2);
                if (pflag != ~0u) flag(pflag);
                pflag = cflag;
                if (u < 8 * NWX) TSW(1);
                if (u < nin && un >= nin) TSW(2);
                if (!more) break;
                if (!early) { WAIT_V(0); flag(pflag); pflag = ~0u; slow(un); }
                u = un;
            }
            WAIT_V(0);
            if (pflag != ~0u) flag(pflag);
            TSW(3);
        }
        if (threadIdx.x == 0) __hip_atomic_store(ctl(me, L_SEQ + b), call, __ATOMIC_RELAXED, __HIP_MEMORY_SCOPE_AGENT);
        return;
    }

    const int role = wi < 16 ? 0 : 1 + ((wi >> 2) & 1);
    const int hi = wi < 16 ? (wi >> 2) & 1 : (wi >> 3) & 1, cb = xcd + 8 * hi;
    const int r4 = wi & 3;
    const unsigned ldsb = (unsigned)(size_t)lds;
    for (int i = threadIdx.x; i < 512; i += 256) ((unsigned*)(lds + LDS_POLL))[i] = 0;
    bool dead = false;
    unsigned call, par;
    {
        const int n = (b - 64) * 4 + wave;
        unsigned raw2[16] = {};
        const Piece p0 = decode_local(n), p1 = decode_local(n + 768);
        piece_load(P, p0, rank, lane, raw);
        piece_load(P, p1, rank, lane, raw2);
        call = rfl(seq0) + 1; par = (call & 1) * PAR;
        piece_xform(p0, raw, o);
        piece_store(P, p0, rank, lane, par, o);
        piece_xform(p1, raw2, o);
        piece_store(P, p1, rank, lane, par, o);
        WAIT_V(4);
        piece_flag(P, p0, rank, lane);
        WAIT_V(0);
        piece_flag(P, p1, rank, lane);
        TS(1);
    }

    f32x16 acc[2][2], acc2[2][2];
#pragma unroll
    for (int m = 0; m < 2; ++m)
#pragma unroll
        for (int i = 0; i < 2; ++i)
#pragma unroll
            for (int e = 0; e < 16; ++e) { acc[m][i][e] = 0.f; acc2[m][i][e] = 0.f; }

    const int step0 = (role == 2 || (role == 0 && hi == 0)) ? 1 : 0, step1 = role == 0 ? 3 : 2;
    int pslot = 0;
    for (int step = step0; step < step1; ++step) {
        unsigned segs; int nseg, wofs = 0, rb;
        if (step == 0) { segs = 0x10; nseg = 2; rb = 12 + r4; wofs = role == 1 ? 8 : 0; }
        else if (step == 2) { rb = 12 + r4; if (hi) { segs = 0x765432; nseg = 6; } else { segs = 0x76543210; nseg = 8; } }
        else if (role == 0) { segs = 0x76543210; nseg = 8; rb = r4; }
        else {
            rb = 4 * role + r4;
            const unsigned lo = (1u << (4 * (4 + r4))) - 1;
            segs = (0x76543210u & lo) | ((0x07654321u) & ~lo); nseg = 7;
        }
        const unsigned* cptr = ctl(me, (lane & 1) ? L_CW + ((((lane >> 1) & 7) + wofs) % 10) * 16 + cb : L_CX + ((lane >> 1) & 7) * 16 + rb);
        const unsigned target = 16 * call;
        const char* gA = me + par + OFF_XS + rb * 65536u + wave * 4096 + lane * 16;
        const char* gB = me + par + OFF_WS + wofs * MiB + cb * 65536u + wave * 4096 + lane * 16;
        const unsigned lw = ldsb + wave * 4096;
        const char* la = lds + (wave >> 1) * 8192 + lane * 16;
        const char* lb = lds + 16384 + (wave & 1) * 8192 + lane * 16;
        unsigned known = 0;
        bool polled = false;
        auto check = [&](int s) {
            const unsigned need = 3u << (2 * s);
            if (polled) {
                unsigned v = ((const unsigned*)(lds + LDS_POLL + pslot * 64))[lane & 15];
                known |= (unsigned)__builtin_amdgcn_ballot_w64(v >= target) & 0xffffu;
                ++pslot; polled = false;
            }
            if ((known & need) != need) {
                unsigned* mw = (unsigned*)(lds + LDS_POLL + pslot * 64);
                if (wave == 0) {
                    unsigned ok = (unsigned)wave_wait(cptr, lane < 16, target, need, tmo, call);
                    if (lane == 0) *mw = ok ? ok : 0x80000000u;
                }
                BAR();
                unsigned ok = rfl(*mw);
                if (ok & 0x80000000u) { dead = true; ok = 0xffffu; }
                known |= ok;
                ++pslot;
            }
            if (s == 2) TS(10 + step);
        };
        bf16x8 fa[2][4], fb[2][4];
#define SB() __builtin_amdgcn_sched_barrier(0)
#define RD(h, slot) do { _Pragma("unroll") for (int k2 = 0; k2 < 2; ++k2) { \
            fa[h][2 * k2] = *(const bf16x8*)(la + (slot) * STAGE_B + (2 * h + k2) * 1024); \
            fb[h][2 * k2] = *(const bf16x8*)(lb + (slot) * STAGE_B + h * 4096 + k2 * 1024); \
            fb[h][2 * k2 + 1] = *(const bf16x8*)(lb + (slot) * STAGE_B + h * 4096 + 2048 + k2 * 1024); \
            fa[h][2 * k2 + 1] = *(const bf16x8*)(la + (slot) * STAGE_B + 4096 + (2 * h + k2) * 1024); } SB(); } while (0)
#define MM(h, k2, m) do { acc[m][0] = __builtin_amdgcn_mfma_f32_32x32x16_bf16(fa[h][2 * k2 + m], fb[h][2 * k2], acc[m][0], 0, 0, 0); \
            acc[m][1] = __builtin_amdgcn_mfma_f32_32x32x16_bf16(fa[h][2 * k2 + m], fb[h][2 * k2 + 1], acc[m][1], 0, 0, 0); SB(); } while (0)
#define MMA(h) do { MM(h, 0, 0); MM(h, 0, 1); MM(h, 1, 0); MM(h, 1, 1); } while (0)
#define MMA_LD(h, g, l) do { const char* g_ = (g); const unsigned l_ = (l); \
            MM(h, 0, 0); glds1k<0>(g_, l_); SB(); MM(h, 0, 1); glds1k<1024>(g_, l_); SB(); \
            MM(h, 1, 0); glds1k<2048>(g_, l_); SB(); MM(h, 1, 1); glds1k<3072>(g_, l_); SB(); } while (0)
        BAR();
        unsigned so = (segs & 15) * MiB;
        check(segs & 15);
#pragma unroll
        for (int kc = 0; kc < 3; ++kc) { glds4k(gA + so + kc * 16384, lw + kc * STAGE_B); glds4k(gB + so + kc * 16384, lw + kc * STAGE_B + 16384); }
        glds4k(gA + so + 3 * 16384, lw + 3 * STAGE_B);
        WAIT_V(20); BAR();
        RD(0, 0);
        for (int gi = 1; gi < nseg; ++gi) {
            segs >>= 4;
            const int sn = segs & 15;
            const unsigned no = sn * MiB;
            RD(1, 0); MMA_LD(0, gB + so + 3 * 16384, lw + 3 * STAGE_B + 16384);
            WAIT_V(16); BAR();
            RD(0, 1); check(sn); SB(); MMA_LD(1, gA + no, lw);
            RD(1, 1); MMA_LD(0, gB + no, lw + 16384);
            WAIT_V(16); BAR();
            RD(0, 2); MMA_LD(1, gA + no + 16384, lw + STAGE_B);
            RD(1, 2);
            if (gi + 1 < nseg && ((known >> (2 * ((segs >> 4) & 15))) & 3) != 3) {
                const unsigned pa = rfl(ldsb + LDS_POLL + pslot * 64);
                if (wave == 0 && lane < 16) glds_word(cptr, pa);
                polled = true;
            }
            SB();
            MMA_LD(0, gB + no + 16384, lw + STAGE_B + 16384);
            WAIT_V(16); BAR();
            RD(0, 3); MMA_LD(1, gA + no + 2 * 16384, lw + 2 * STAGE_B);
            RD(1, 3); MMA_LD(0, gB + no + 2 * 16384, lw + 2 * STAGE_B + 16384);
            WAIT_V(16); BAR();
            RD(0, 0); MMA_LD(1, gA + no + 3 * 16384, lw + 3 * STAGE_B);
            so = no;
        }
        RD(1, 0); MMA_LD(0, gB + so + 3 * 16384, lw + 3 * STAGE_B + 16384);
        WAIT_V(16); BAR();
        RD(0, 1); MMA(1);
        RD(1, 1); MMA(0);
        WAIT_V(8); BAR();
        RD(0, 2); MMA(1);
        RD(1, 2); MMA(0);
        WAIT_V(0); BAR();
        RD(0, 3); MMA(1);
        RD(1, 3); MMA(0);
        MMA(1);
        TS(4 + step);
        if (step == 1) {
            const int d = role == 0 ? 1 : 4 + r4, tile = rb * 16 + cb;
            unsigned pos = 0;
            if (threadIdx.x == 0) pos = __hip_atomic_fetch_add(ctl(me, L_QT + d), 1u, __ATOMIC_RELAXED, __HIP_MEMORY_SCOPE_AGENT);
            if (dead) poison(acc);
            __amdgpu_buffer_rsrc_t rs = rsrc(me, OFF_CTL);
#pragma unroll
            for (int q = 0; q < 8; ++q) {
                u32x4 v;
#pragma unroll
                for (int j = 0; j < 4; ++j) v[j] = pack2(acc[q >> 2][0][4 * (q & 3) + j], acc[q >> 2][1][4 * (q & 3) + j]);
                __builtin_amdgcn_raw_buffer_store_b128(v, rs, lane * 16 + q * 1024, OFF_LP + tile * 32768u + wave * 8192, AUX_SYS);
            }
            WAIT_V(0);
            BAR();
            if (threadIdx.x == 0)
                __hip_atomic_store(ctl(me, L_Q + 2 * d) + ((pos - (call - 1) * (role == 0 ? 64 : 32)) & 63), (call << 8) | tile, __ATOMIC_RELAXED, __HIP_MEMORY_SCOPE_AGENT);
            TS(7);
        }
        if (step < 2) {
#pragma unroll
            for (int m = 0; m < 2; ++m)
#pragma unroll
                for (int i = 0; i < 2; ++i) {
                    const f32x16 t = acc2[m][i];
                    acc2[m][i] = acc[m][i];
                    acc[m][i] = t;
                }
        }
    }

    if (role != 2) {
        __amdgpu_buffer_rsrc_t rs = rsrc(me, OFF_LP);
        const int far = role, tile = r4 * 16 + cb;
        if (!wave_wait(ctl(me, L_FP + (far + (lane & 1)) * 64 + tile), lane <= far, 8 * call, far ? 3ull : 1ull, tmo, call)) dead = true;
        TS(8);
        u32x4 pv[2][8] = {};
#pragma unroll
        for (int q = 0; q < 8; ++q)
            pv[0][q] = __builtin_amdgcn_raw_buffer_load_b128(rs, lane * 16 + q * 1024, par + OFF_PS + (far * 64 + tile) * 32768u + wave * 8192, AUX_SYS);
        if (far) {
#pragma unroll
            for (int q = 0; q < 8; ++q)
                pv[1][q] = __builtin_amdgcn_raw_buffer_load_b128(rs, lane * 16 + q * 1024, par + OFF_PS + (128 + tile) * 32768u + wave * 8192, AUX_SYS);
        }
        if (rfl(__hip_atomic_load(tmo, __ATOMIC_RELAXED, __HIP_MEMORY_SCOPE_AGENT)) == (int)call) dead = true;
        if (dead) poison(acc);
        unsigned* op = (unsigned*)(P.out + (size_t)(128 * r4 + 64 * (wave >> 1) + 4 * (lane >> 5)) * 4096
                                   + 2048 * ((rank >> 2) ^ far) + 128 * cb + 64 * (wave & 1) + 2 * (lane & 31));
#pragma unroll
        for (int q = 0; q < 8; ++q)
#pragma unroll
            for (int j = 0; j < 4; ++j) {
                float y0 = acc[q >> 2][0][4 * (q & 3) + j], y1 = acc[q >> 2][1][4 * (q & 3) + j];
#pragma unroll
                for (int s = 0; s < 2; ++s) {
                    y0 += __builtin_bit_cast(float, pv[s][q][j] << 16);
                    y1 += __builtin_bit_cast(float, pv[s][q][j] & 0xffff0000u);
                }
                op[(size_t)(32 * (q >> 2) + 8 * (q & 3) + j) * 2048] = pack2(gelu(y0), gelu(y1));
            }
        TS(9);
    }
    if (threadIdx.x == 0) __hip_atomic_store(ctl(me, L_SEQ + b), call, __ATOMIC_RELAXED, __HIP_MEMORY_SCOPE_AGENT);
}

extern "C" void kernel_launch(int rank, int nranks, ncclComm_t comm,
                              hipStream_t stream,
                              void* const* d_in, const int* in_sizes, int n_in,
                              void* d_out, int out_size,
                              const int* in_dtypes, int out_dtype,
                              void* const* d_out_all,
                              void* const* d_ws_all, size_t ws_bytes) {
    if (nranks != NR || (unsigned)rank >= (unsigned)NR || n_in != 2 || in_sizes[0] != 4096 * 512 || in_sizes[1] != 512 * 4096 ||
        out_size != 512 * 4096 || ws_bytes < WS_NEED) {
        fprintf(stderr, "kernel_launch: unexpected problem shape\n");
        return;
    }
    Params P;
    P.x = (const uint16_t*)d_in[0]; P.w = (const uint16_t*)d_in[1]; P.out = (uint16_t*)d_out;
    for (int p = 0; p < NR; ++p) P.ws[p] = (char*)d_ws_all[p];
    P.rank = rank;
    fused_gemm_rs<<<256, 256, 0, stream>>>(P);
    HIP_CHECK(hipGetLastError());
}
```

```cpp
#include <hip/hip_runtime.h>
#include <rccl/rccl.h>
#include <cstdio>
#include <stdint.h>

#define HIP_CHECK(expr) do { hipError_t _e = (expr); if (_e != hipSuccess) { \
    fprintf(stderr, "HIP error %s at %s:%d\n", hipGetErrorString(_e), __FILE__, __LINE__); \
    return; } } while (0)

typedef short bf16x8 __attribute__((ext_vector_type(8)));
typedef float f32x16 __attribute__((ext_vector_type(16)));
typedef unsigned u32x4 __attribute__((ext_vector_type(4)));
typedef __bf16 bf16x2 __attribute__((ext_vector_type(2)));
typedef float f32x2 __attribute__((ext_vector_type(2)));
#define DI __device__ __forceinline__

constexpr int NR = 8;
constexpr unsigned MiB = 1u << 20;
constexpr unsigned OFF_XS = 0;
constexpr unsigned OFF_WS = 8 * MiB;
constexpr unsigned OFF_PS = 18 * MiB;
constexpr unsigned OFF_RI = 24 * MiB;
constexpr unsigned PAR = 25 * MiB;
constexpr unsigned OFF_LP = 2 * PAR;
constexpr unsigned OFF_CTL = OFF_LP + 8 * MiB;
constexpr unsigned L_CX = 0;
constexpr unsigned L_CW = L_CX + 128;
constexpr unsigned L_FP = L_CW + 160;
constexpr unsigned L_SEQ = L_FP + 192;
constexpr unsigned L_TMO = L_SEQ + 256;
constexpr unsigned L_QT = L_TMO + 1;
constexpr unsigned L_Q = L_QT + 8;
constexpr unsigned L_TK = L_Q + 16;
constexpr unsigned L_RF = L_TK + 8;
constexpr unsigned L_END = L_RF + 16;
constexpr unsigned OFF_DBG = OFF_CTL + 512 * 1024;
constexpr unsigned WS_NEED = OFF_CTL + MiB;
static_assert(L_END * 128 <= 512 * 1024, "ctl");
#define LINK_LIST 1, 2, 3, 4, 5, 6, 7, 1, 2, 3, 4, 5, 6, 7, 0, 0, 0, 0, 0, 0, 0, 0, 0, 0, 0, 0, 0, 0, 0, 0, 0, 0
constexpr unsigned char LINKS[32] = {LINK_LIST};
constexpr unsigned long long lm16(int h, int idx) {
    unsigned long long v = 0;
    for (int i = 0; i < 16; ++i) { int c = 0; for (int j = 0; j < 16 * h + i; ++j) c += LINKS[j] == LINKS[16 * h + i]; v |= (unsigned long long)(idx ? (c & 7) : LINKS[16 * h + i]) << (4 * i); }
    return v;
}
constexpr unsigned lmcnt() { unsigned v = 0; for (int i = 0; i < 32; ++i) if (LINKS[i]) v += 1u << (4 * LINKS[i]); return v; }
#define RC0 9
#define RCN 2
#define MAPV 1
constexpr unsigned long long DEADLINE = 100000000ull;
constexpr int AUX_SYS = 17;

constexpr int RING = 4, STAGE_B = 32768, LDS_POLL = RING * STAGE_B, LDS_BYTES = LDS_POLL + 2048;

struct Params {
    const uint16_t* x; const uint16_t* w; uint16_t* out;
    char* ws[NR];
    int rank;
};

#define TS(i) do {} while (0)
#define TSW(i) do {} while (0)

DI unsigned* ctl(char* ws, unsigned line) { return (unsigned*)(ws + OFF_CTL + line * 128u); }
DI __amdgpu_buffer_rsrc_t rsrc(const void* p, unsigned bytes) { return __builtin_amdgcn_make_buffer_rsrc((void*)p, 0, bytes, 0x00020000); }
DI int rfl(int v) { return __builtin_amdgcn_readfirstlane(v); }
#define WAIT_V(n) asm volatile("s_waitcnt vmcnt(" #n ")" ::: "memory")
#define BAR() do { asm volatile("s_waitcnt lgkmcnt(0)" ::: "memory"); __builtin_amdgcn_s_barrier(); asm volatile("" ::: "memory"); } while (0)

#define LDFL " sc0 sc1"
DI void glds4k(const char* g, unsigned lds) {
    unsigned keep;
    asm volatile("s_mov_b32 %0, m0\n\ts_mov_b32 m0, %2\n\ts_nop 0\n\t"
                 "global_load_lds_dwordx4 %1, off" LDFL "\n\t"
                 "global_load_lds_dwordx4 %1, off offset:1024" LDFL "\n\t"
                 "global_load_lds_dwordx4 %1, off offset:2048" LDFL "\n\t"
                 "global_load_lds_dwordx4 %1, off offset:3072" LDFL "\n\t"
                 "s_mov_b32 m0, %0" : "=&s"(keep) : "v"(g), "s"(lds) : "memory");
}
template <int OFS> DI void glds1k(const char* g, unsigned lds) {
    unsigned keep;
    asm volatile("s_mov_b32 %0, m0\n\ts_mov_b32 m0, %2\n\ts_nop 0\n\t"
                 "global_load_lds_dwordx4 %1, off offset:%c3" LDFL "\n\t"
                 "s_mov_b32 m0, %0" : "=&s"(keep) : "v"(g), "s"(lds), "i"(OFS) : "memory");
}
DI void glds4k_s(const char* g, unsigned v, unsigned lds) {
    unsigned keep;
    asm volatile("s_mov_b32 %0, m0\n\ts_mov_b32 m0, %3\n\ts_nop 0\n\t"
                 "global_load_lds_dwordx4 %1, %2 sc0 sc1\n\t"
                 "global_load_lds_dwordx4 %1, %2 offset:1024 sc0 sc1\n\t"
                 "global_load_lds_dwordx4 %1, %2 offset:2048 sc0 sc1\n\t"
                 "global_load_lds_dwordx4 %1, %2 offset:3072 sc0 sc1\n\t"
                 "s_mov_b32 m0, %0" : "=&s"(keep) : "v"(v), "s"(g), "s"(lds) : "memory");
}
DI void glds4k_x(const char* g, unsigned v, unsigned lds) {
    unsigned keep;
    asm volatile("s_mov_b32 %0, m0\n\ts_mov_b32 m0, %3\n\ts_nop 0\n\t"
                 "global_load_lds_dwordx4 %1, %2\n\ts_add_u32 m0, m0, 992\n\ts_nop 0\n\t"
                 "global_load_lds_dwordx4 %1, %2 offset:32\n\ts_add_u32 m0, m0, 992\n\ts_nop 0\n\t"
                 "global_load_lds_dwordx4 %1, %2 offset:64\n\ts_add_u32 m0, m0, 992\n\ts_nop 0\n\t"
                 "global_load_lds_dwordx4 %1, %2 offset:96\n\t"
                 "s_mov_b32 m0, %0" : "=&s"(keep) : "v"(v), "s"(g), "s"(lds) : "memory", "scc");
}
DI void glds_word(const unsigned* g, unsigned lds) {
    unsigned keep;
    asm volatile("s_mov_b32 %0, m0\n\ts_mov_b32 m0, %2\n\ts_nop 0\n\t"
                 "global_load_lds_dword %1, off sc0 sc1\n\t"
                 "s_mov_b32 m0, %0" : "=&s"(keep) : "v"(g), "s"(lds) : "memory");
}

DI unsigned long long wave_wait(const unsigned* p, bool active, unsigned target, unsigned long long need, unsigned* tmo, unsigned call) {
    unsigned long long t0 = __builtin_amdgcn_s_memrealtime();
    for (unsigned it = 0;; ++it) {
        unsigned v = 0;
        if (active) v = __hip_atomic_load(p, __ATOMIC_RELAXED, __HIP_MEMORY_SCOPE_SYSTEM);
        unsigned long long ok = __builtin_amdgcn_ballot_w64(active && v >= target);
        if (ok & (ok >> 1) & need) return ok;
        if ((it & 63) == 63) {
            unsigned t = rfl(__hip_atomic_load(tmo, __ATOMIC_RELAXED, __HIP_MEMORY_SCOPE_AGENT));
            if (t == call || __builtin_amdgcn_s_memrealtime() - t0 > DEADLINE) {
                __hip_atomic_store(tmo, call, __ATOMIC_RELAXED, __HIP_MEMORY_SCOPE_AGENT);
                return 0;
            }
        }
        __builtin_amdgcn_s_sleep(4);
    }
}

DI int brick_row(int r, int rb) {
    if (rb < 4) return 512 * (r ^ 1) + 128 * rb;
    if (rb < 12) return 512 * (r ^ (4 + (rb & 3))) + 256 * ((r >> 1) & 1) + 128 * ((rb - 4) >> 2);
    return 512 * r + 128 * (rb - 12);
}

#define ORDER_LIST 0, 1, 16, 17, 2, 18, 3, 19, 20, 4, 21, 5, 22, 6, 23, 7, 24, 8, 25, 9, 26, 10, 27, 11, 28, 29, 30, 31, 12, 13, 14, 15
constexpr unsigned char ORDER[32] = {ORDER_LIST};
constexpr unsigned long long ord8(int g) { unsigned long long v = 0; for (int i = 0; i < 8; ++i) v |= (unsigned long long)ORDER[8 * g + i] << (8 * i); return v; }
constexpr unsigned long long pos8() { unsigned long long v = 0; for (int q = 0; q < 32; ++q) if (ORDER[q] >= 4 && ORDER[q] < 12) v |= (unsigned long long)q << (8 * (ORDER[q] - 4)); return v; }
DI int order_at(int q) {
    const unsigned long long v = (q & 16) ? ((q & 8) ? ord8(3) : ord8(2)) : ((q & 8) ? ord8(1) : ord8(0));
    return (int)(v >> (8 * (q & 7))) & 31;
}
struct Piece { int kind; int seg; int blk; int sub; };
DI Piece decode_local(int n) {
    Piece p;
    int rnd = n >= 768; n -= rnd * 768;
    p.sub = n & 15;
    if (n < 512) { p.kind = 1; p.seg = rnd * 8 + (n >> 8); p.blk = (n >> 4) & 15; }
    else { n -= 512; p.kind = 0; p.seg = n >> 7; int q = (n >> 4) & 7; p.blk = rnd ? 4 + q : (q < 4 ? q : q + 8); }
    return p;
}
DI void piece_load(const Params& P, const Piece& p, int dst, int lane, unsigned (&r)[16]) {
    const int k0 = 256 * ((p.seg >= 2 && p.seg < 8) ? (dst & 1) : (p.seg & 1)) + 64 * (p.sub >> 2);
    if (p.kind == 0) {
        const int row = brick_row(dst, p.blk) + 32 * (p.sub & 3) + (lane & 31);
        const u32x4* s = (const u32x4*)(P.x + (size_t)row * 512 + k0 + 8 * (lane >> 5));
#pragma unroll
        for (int ks = 0; ks < 4; ++ks) { u32x4 v = s[2 * ks]; r[4 * ks] = v[0]; r[4 * ks + 1] = v[1]; r[4 * ks + 2] = v[2]; r[4 * ks + 3] = v[3]; }
    } else {
        const int col = 2048 * ((dst >> 2) ^ (p.seg >> 3)) + 128 * p.blk + 64 * ((p.sub >> 1) & 1) + 2 * (lane & 31);
        const int k = k0 + 32 * (p.sub & 1) + 8 * (lane >> 5);
        const unsigned* s = (const unsigned*)(P.w + (size_t)k * 4096 + col);
#pragma unroll
        for (int ks = 0; ks < 2; ++ks)
#pragma unroll
            for (int j = 0; j < 8; ++j) r[8 * ks + j] = s[(16 * ks + j) * 2048];
    }
}
DI int ps_index(int r, int t) {
    const int rb = t >> 4, cb = t & 15;
    return rb < 4 ? t : (1 + (r & 1)) * 64 + (2 * ((r >> 1) & 1) + ((rb - 4) >> 2)) * 16 + cb;
}
DI void piece_xform(const Piece& p, const unsigned (&r)[16], u32x4 (&o)[4]) {
    if (p.kind == 1) {
#pragma unroll
        for (int ks = 0; ks < 2; ++ks)
#pragma unroll
            for (int m = 0; m < 4; ++m) {
                o[ks][m] = __builtin_amdgcn_perm(r[8 * ks + 2 * m + 1], r[8 * ks + 2 * m], 0x05040100u);
                o[2 + ks][m] = __builtin_amdgcn_perm(r[8 * ks + 2 * m + 1], r[8 * ks + 2 * m], 0x07060302u);
            }
    } else {
#pragma unroll
        for (int ks = 0; ks < 4; ++ks) o[ks] = u32x4{r[4 * ks], r[4 * ks + 1], r[4 * ks + 2], r[4 * ks + 3]};
    }
    asm volatile("" : "+v"(o[0]), "+v"(o[1]), "+v"(o[2]), "+v"(o[3]) :: "memory");
}
DI void piece_store(const Params& P, const Piece& p, int dst, int lane, unsigned par, const u32x4 (&o)[4]) {
    __amdgpu_buffer_rsrc_t rs = rsrc(P.ws[dst & 7], OFF_LP);
    const unsigned off = par + p.sub * 4096u + (p.kind ? OFF_WS : OFF_XS) + p.seg * MiB + p.blk * 65536u;
#pragma unroll
    for (int q = 0; q < 4; ++q) __builtin_amdgcn_raw_buffer_store_b128(o[q], rs, lane * 16 + q * 1024, off, AUX_SYS);
}
DI void piece_flag(const Params& P, const Piece& p, int dst, int lane) {
    const unsigned line = (p.kind ? L_CW : L_CX) + p.seg * 16 + p.blk;
    if (lane == 0) __hip_atomic_fetch_add(ctl(P.ws[dst & 7], line), 1u, __ATOMIC_RELAXED, __HIP_MEMORY_SCOPE_SYSTEM);
}

DI unsigned pack2(float a, float b) { f32x2 v = {a, b}; return __builtin_bit_cast(unsigned, __builtin_convertvector(v, bf16x2)); }
DI f32x2 gelu2(f32x2 y) {
    const float c2 = -2.0f * 0.7978845608028654f * 1.4426950408889634f;
    const f32x2 t = (y * y * (0.044715f * c2) + c2) * y;
    const f32x2 d = f32x2{__builtin_amdgcn_exp2f(t[0]), __builtin_amdgcn_exp2f(t[1])} + 1.0f;
    return y * f32x2{__builtin_amdgcn_rcpf(d[0]), __builtin_amdgcn_rcpf(d[1])};
}

DI void poison(f32x16 (&a)[2][2]) {
#pragma unroll
    for (int m = 0; m < 2; ++m)
#pragma unroll
        for (int i = 0; i < 2; ++i)
#pragma unroll
            for (int e = 0; e < 16; ++e) a[m][i][e] = __builtin_nanf("");
}

__global__ __launch_bounds__(256, 1) void fused_gemm_rs(Params P) {
    __shared__ __attribute__((aligned(1024))) char lds[LDS_BYTES];
    const int b = blockIdx.x, lane = threadIdx.x & 63, wave = rfl(threadIdx.x >> 6);
    const int xcd = b & 7, wi = b >> 3;
    const int rank = P.rank;
    char* const me = P.ws[rank & 7];
    TS(0);
    const unsigned seq0 = __hip_atomic_load(ctl(me, L_SEQ + b), __ATOMIC_RELAXED, __HIP_MEMORY_SCOPE_AGENT);
    unsigned* const tmo = ctl(me, L_TMO);
    unsigned raw[16] = {};
    u32x4 o[4];

    const unsigned ldsb = (unsigned)(size_t)lds;
    if (wi < 8) {
        const int i = wave * 8 + wi;
        const int d = (int)((i & 16 ? lm16(1, 0) : lm16(0, 0)) >> (4 * (i & 15))) & 7, dst = rank ^ d;
        const int nw = 8 * ((lmcnt() >> (4 * d)) & 15);
        const int nin = d == 1 ? 0 : (d < 4 ? 512 : 480);
        const int nend = d == 0 ? 0 : nin + (d == 1 ? 512 - 32 * RCN : (d < 4 ? 32 * RCN : 256));
        int sk1 = 99, sk2 = 99;
        if (d >= 4) { const int a = (pos8() >> (8 * (d - 4))) & 31, c = (pos8() >> (8 * d)) & 31; sk1 = a < c ? a : c; sk2 = a < c ? c : a; }
        const int lseg = (d >> 2) * 8 + (dst & 1);
        const unsigned vx = (lane & 31) * 1024 + (lane >> 5) * 16, vl = lane * 16;
        const __amdgpu_buffer_rsrc_t rd = rsrc(P.ws[dst & 7], OFF_LP);
        const unsigned lb = ldsb + wave * 32768;
        unsigned* const snap = (unsigned*)(lds + wave * 32768 + 16384);
        const unsigned* const rp = (d == 1 || lane < 32) ? ctl(me, L_Q + 2 * d) + lane
                                 : ctl(me, lane < 48 ? L_CW + lseg * 16 + (lane & 15) : L_RF + ((d & 1) ^ 1) * 8 + (lane & 7));
        unsigned* const tkp = ctl(me, L_TK + d);
        unsigned call = 0, par = 0, ent = 0, d2 = 0, f2 = 0, one = 1;
        int ws = 0;
        auto fetch = [&](int u) -> bool {
            const unsigned l = rfl(lb + (ws & 3) * 4096);
            if (u >= nin) {
                const int qi = ((u - nin) >> 3) & 63, sub = (u - nin) & 7;
                const bool side = (d & 6) == 2;
                if (side && qi >= 2 * RCN) {
                    const int sl = (qi - 2 * RCN) & 7, t = (2 * (sl & 1) + ((d & 1) ^ 1)) * 16 + RC0 + (sl >> 1);
                    if ((unsigned)__builtin_amdgcn_readlane(ent, 48 + sl) < 8 * call) return false;
                    d2 = OFF_PS + t * 32768u + sub * 4096u; f2 = L_FP + t;
                    glds4k_s(me + par + OFF_RI + (((d & 1) ^ 1) * 8 + sl) * 32768u + sub * 4096u, vl, l);
                } else {
                    const unsigned e = __builtin_amdgcn_readlane(ent, qi);
                    if (e < call << 8) return false;
                    const int t = (e & 255) < 192 ? (e & 255) : 0, pi = ps_index(rank, t), ri = (d & 1) * 8 + (qi & 7);
                    d2 = (side ? OFF_RI + ri * 32768u : OFF_PS + pi * 32768u) + sub * 4096u; f2 = side ? L_RF + ri : L_FP + pi;
                    glds4k_s(me + OFF_LP + t * 32768u + sub * 4096u, vl, l);
                }
            } else {
                int q = u >> 4; q += q >= sk1; q += q >= sk2;
                const int bk = order_at(q & 31), blk = bk & 15, sub = u & 15;
                if (bk >= 16) {
                    if ((unsigned)__builtin_amdgcn_readlane(ent, 32 + blk) < 16 * call) return false;
                    d2 = OFF_WS + d * MiB + blk * 65536u + sub * 4096u; f2 = L_CW + d * 16 + blk;
                    glds4k_s(me + par + OFF_WS + lseg * MiB + blk * 65536u + sub * 4096u, vl, l);
                } else {
                    d2 = OFF_XS + d * MiB + blk * 65536u + sub * 4096u; f2 = L_CX + d * 16 + blk;
                    glds4k_x((const char*)P.x + (brick_row(dst, blk) + 32 * (sub & 3)) * 1024 + (dst & 1) * 512 + (sub >> 2) * 128, vx, l);
                }
            }
            ++ws;
            return true;
        };
        const int u0 = (int)(((i & 16 ? lm16(1, 1) : lm16(0, 1)) >> (4 * (i & 15))) & 7) * 8 + xcd;
        bool v0 = false, v1 = false;
        unsigned d0 = 0, f0 = 0, d1 = 0, f1 = 0, tq = 0;
        int tk = u0, tl = 0;
        auto claim = [&]() {
            asm volatile("s_mov_b32 exec_lo, %3\n\ts_mov_b32 exec_hi, 0\n\tglobal_atomic_add %0, %1, %2, off sc0 sc1\n\ts_mov_b64 exec, -1"
                         : "+v"(tq) : "v"(tkp), "v"(one), "s"(rfl(1 + tl)) : "memory");
            tl ^= 1;
        };
        if (d > 1) {
            fetch(u0); claim();
            v0 = true; d0 = d2; f0 = f2; tk = u0 + nw;
        }
        snap[lane] = 0; snap[64 + lane] = 0;
        call = rfl(seq0) + 1; par = (call & 1) * PAR;
        const unsigned tofs = 2 * nw - (call - 1) * nend;
        WAIT_V(0);
        if (d == 1) tq = u0 + nw - tofs;
        int sp = 0, rs = 0;
        unsigned pf = ~0u, cf = ~0u, idle = 0;
        const unsigned long long t0 = __builtin_amdgcn_s_memrealtime();
        while (nend) {
            const bool S = v0, F = pf != ~0u;
            const unsigned sv = snap[sp * 64 + lane];
            if (S) {
#pragma unroll
                for (int q = 0; q < 4; ++q) o[q] = *(const u32x4*)(lds + wave * 32768 + (rs & 3) * 4096 + q * 1024 + vl);
                ++rs;
            }
            if (F) asm volatile("s_mov_b64 exec, 1\n\tglobal_atomic_add %0, %1, off sc1\n\ts_mov_b64 exec, -1" :: "v"(ctl(P.ws[dst & 7], pf)), "v"(one) : "memory");
            if (!S && !v1 && cf == ~0u && tk >= nend) break;
            ent = sv > ent ? sv : ent;
            if (S) {
#pragma unroll
                for (int q = 0; q < 4; ++q) __builtin_amdgcn_raw_buffer_store_b128(o[q], rd, vl + par + d0 + q * 1024, 0, AUX_SYS);
            }
            const bool E = S && f0 - L_FP < 192u;
            if (E) asm volatile("s_mov_b64 exec, 1\n\tglobal_atomic_add %0, %1, off sc1\n\ts_mov_b64 exec, -1" :: "v"(ctl(P.ws[dst & 7], f0)), "v"(one) : "memory");
            asm volatile("" : "+v"(ent) :: "memory");
            const bool L = tk < nend && fetch(tk);
            if (L) claim();
            glds_word(rp, rfl(lb + 16384 + sp * 256));
            if (!S) WAIT_V(0);
            else if (L) { if (F && E) WAIT_V(12); else if (F || E) WAIT_V(11); else WAIT_V(10); }
            else { if (F && E) WAIT_V(7); else if (F || E) WAIT_V(6); else WAIT_V(5); }
            asm volatile("" : "+v"(tq) :: "memory");
            pf = cf; cf = S && !E ? f0 : ~0u;
            v0 = v1; d0 = d1; f0 = f1;
            v1 = L; d1 = d2; f1 = f2;
            if (S) sp ^= 1;
            else if (!v0) { v0 = v1; d0 = d1; f0 = f1; v1 = false; }
            if (L) tk = (int)(__builtin_amdgcn_readlane(tq, rfl(tl)) + tofs);
            else if (!S && (++idle & 63) == 0) {
                const unsigned t = rfl(__hip_atomic_load(tmo, __ATOMIC_RELAXED, __HIP_MEMORY_SCOPE_AGENT));
                if (t == call || __builtin_amdgcn_s_memrealtime() - t0 > DEADLINE) {
                    __hip_atomic_store(tmo, call, __ATOMIC_RELAXED, __HIP_MEMORY_SCOPE_AGENT);
                    break;
                }
            }
        }
        TSW(3);
        if (threadIdx.x == 0) __hip_atomic_store(ctl(me, L_SEQ + b), call, __ATOMIC_RELAXED, __HIP_MEMORY_SCOPE_AGENT);
        return;
    }

    const int grp = (wi - 8) >> 2, own = grp >= 4 ? 0 : 1 + (grp & 1);
    const int r4 = xcd & 3, cb = 8 * (grp & 1) + 2 * (wi & 3) + (xcd >> 2);
    const int ro = 2 * (xcd & 1) + ((grp >> 1) & 1), co = 4 * (xcd & 2) + 2 * (wi & 3) + (xcd >> 2);
    for (int i = threadIdx.x; i < 512; i += 256) ((unsigned*)(lds + LDS_POLL))[i] = 0;
    bool dead = false;
    unsigned call, par;
    {
        const int n = (b - 64) * 4 + wave;
        unsigned raw2[16] = {};
        const Piece p0 = decode_local(n), p1 = decode_local(n + 768);
        piece_load(P, p0, rank, lane, raw);
        piece_load(P, p1, rank, lane, raw2);
        call = rfl(seq0) + 1; par = (call & 1) * PAR;
        piece_xform(p0, raw, o);
        piece_store(P, p0, rank, lane, par, o);
        piece_xform(p1, raw2, o);
        piece_store(P, p1, rank, lane, par, o);
        WAIT_V(4);
        piece_flag(P, p0, rank, lane);
        WAIT_V(0);
        piece_flag(P, p1, rank, lane);
        TS(1);
    }

    f32x16 acc[2][2], acc2[2][2];
#pragma unroll
    for (int m = 0; m < 2; ++m)
#pragma unroll
        for (int i = 0; i < 2; ++i)
#pragma unroll
            for (int e = 0; e < 16; ++e) { acc[m][i][e] = 0.f; acc2[m][i][e] = 0.f; }

    const int step0 = own == 2 ? 0 : 1, step1 = own == 1 ? 3 : 2;
    int pslot = 0;
    for (int step = step0; step < step1; ++step) {
        unsigned rem = 0x5555; int wofs = 0, rb = 12 + ro;
        const int cc = step == 1 ? cb : co;
        if (step == 0) { rem = 5; wofs = 8; }
        else if (step == 1) {
            rb = 4 * (grp >> 1) + r4;
            if (grp >= 2) rem &= ~(0x100u << (2 * r4));
        }
        const unsigned* cptr = ctl(me, (lane & 1) ? L_CW + ((((lane >> 1) & 7) + wofs) % 10) * 16 + cc : L_CX + ((lane >> 1) & 7) * 16 + rb);
        const unsigned target = 16 * call;
        const char* gA = me + par + OFF_XS + rb * 65536u + wave * 4096 + lane * 16;
        const char* gB = me + par + OFF_WS + wofs * MiB + cc * 65536u + wave * 4096 + lane * 16;
        const unsigned lw = ldsb + wave * 4096;
        const char* la = lds + (wave >> 1) * 8192 + lane * 16;
        const char* lb = lds + 16384 + (wave & 1) * 8192 + lane * 16;
        unsigned known = 0;
        bool polled = false;
        auto next = [&]() -> int {
            if (polled) {
                unsigned v = ((const unsigned*)(lds + LDS_POLL + pslot * 64))[lane & 15];
                known |= (unsigned)__builtin_amdgcn_ballot_w64(v >= target) & 0xffffu;
                ++pslot; polled = false;
            }
            if (!(known & (known >> 1) & rem)) {
                unsigned* mw = (unsigned*)(lds + LDS_POLL + pslot * 64);
                if (wave == 0) {
                    unsigned ok = (unsigned)wave_wait(cptr, lane < 16, target, rem, tmo, call);
                    if (lane == 0) *mw = ok ? ok : 0x80000000u;
                }
                BAR();
                unsigned ok = rfl(*mw);
                if (ok & 0x80000000u) { dead = true; ok = 0xffffu; }
                known |= ok;
                ++pslot;
            }
            const int s2 = __builtin_ctz(known & (known >> 1) & rem);
            rem &= ~(1u << s2);
            return s2 >> 1;
        };
        bf16x8 fa[2][4], fb[2][4];
#define SB() __builtin_amdgcn_sched_barrier(0)
#define RD(h, slot) do { _Pragma("unroll") for (int k2 = 0; k2 < 2; ++k2) { \
            fa[h][2 * k2] = *(const bf16x8*)(la + (slot) * STAGE_B + (2 * h + k2) * 1024); \
            fb[h][2 * k2] = *(const bf16x8*)(lb + (slot) * STAGE_B + h * 4096 + k2 * 1024); \
            fb[h][2 * k2 + 1] = *(const bf16x8*)(lb + (slot) * STAGE_B + h * 4096 + 2048 + k2 * 1024); \
            fa[h][2 * k2 + 1] = *(const bf16x8*)(la + (slot) * STAGE_B + 4096 + (2 * h + k2) * 1024); } SB(); } while (0)
#define MM(h, k2, m) do { acc[m][0] = __builtin_amdgcn_mfma_f32_32x32x16_bf16(fa[h][2 * k2 + m], fb[h][2 * k2], acc[m][0], 0, 0, 0); \
            acc[m][1] = __builtin_amdgcn_mfma_f32_32x32x16_bf16(fa[h][2 * k2 + m], fb[h][2 * k2 + 1], acc[m][1], 0, 0, 0); SB(); } while (0)
#define MMA(h) do { MM(h, 0, 0); MM(h, 0, 1); MM(h, 1, 0); MM(h, 1, 1); } while (0)
#define MMA_LD(h, g, l) do { const char* g_ = (g); const unsigned l_ = (l); \
            MM(h, 0, 0); glds1k<0>(g_, l_); SB(); MM(h, 0, 1); glds1k<1024>(g_, l_); SB(); \
            MM(h, 1, 0); glds1k<2048>(g_, l_); SB(); MM(h, 1, 1); glds1k<3072>(g_, l_); SB(); } while (0)
        BAR();
        unsigned so = next() * MiB;
#pragma unroll
        for (int kc = 0; kc < 3; ++kc) { glds4k(gA + so + kc * 16384, lw + kc * STAGE_B); glds4k(gB + so + kc * 16384, lw + kc * STAGE_B + 16384); }
        glds4k(gA + so + 3 * 16384, lw + 3 * STAGE_B);
        WAIT_V(20); BAR();
        RD(0, 0);
        while (rem) {
            RD(1, 0); MMA_LD(0, gB + so + 3 * 16384, lw + 3 * STAGE_B + 16384);
            WAIT_V(16); BAR();
            RD(0, 1);
            const unsigned no = next() * MiB;
            SB(); MMA_LD(1, gA + no, lw);
            RD(1, 1); MMA_LD(0, gB + no, lw + 16384);
            WAIT_V(16); BAR();
            RD(0, 2); MMA_LD(1, gA + no + 16384, lw + STAGE_B);
            RD(1, 2);
            if (rem && !(known & (known >> 1) & rem)) {
                const unsigned pa = rfl(ldsb + LDS_POLL + pslot * 64);
                if (wave == 0 && lane < 16) glds_word(cptr, pa);
                polled = true;
            }
            SB();
            MMA_LD(0, gB + no + 16384, lw + STAGE_B + 16384);
            WAIT_V(16); BAR();
            RD(0, 3); MMA_LD(1, gA + no + 2 * 16384, lw + 2 * STAGE_B);
            RD(1, 3); MMA_LD(0, gB + no + 2 * 16384, lw + 2 * STAGE_B + 16384);
            WAIT_V(16); BAR();
            RD(0, 0); MMA_LD(1, gA + no + 3 * 16384, lw + 3 * STAGE_B);
            so = no;
        }
        RD(1, 0); MMA_LD(0, gB + so + 3 * 16384, lw + 3 * STAGE_B + 16384);
        WAIT_V(16); BAR();
        RD(0, 1); MMA(1);
        RD(1, 1); MMA(0);
        WAIT_V(8); BAR();
        RD(0, 2); MMA(1);
        RD(1, 2); MMA(0);
        WAIT_V(0); BAR();
        RD(0, 3); MMA(1);
        RD(1, 3); MMA(0);
        MMA(1);
        TS(4 + step);
        if (step == 1) {
            const bool relay = grp < 2 && (unsigned)(cb - RC0) < RCN;
            const int d = relay ? 2 + (r4 & 1) : (grp < 2 ? 1 : 4 + r4), tile = rb * 16 + cb;
            unsigned pos = (cb - RC0) * 2 + (r4 >> 1);
            if (threadIdx.x == 0 && !relay)
                pos = __hip_atomic_fetch_add(ctl(me, L_QT + d), 1u, __ATOMIC_RELAXED, __HIP_MEMORY_SCOPE_AGENT) - (call - 1) * (grp < 2 ? 64 - 4 * RCN : 32);
            if (dead) poison(acc);
            __amdgpu_buffer_rsrc_t rs = rsrc(me, OFF_CTL);
#pragma unroll
            for (int q = 0; q < 8; ++q) {
                u32x4 v;
#pragma unroll
                for (int j = 0; j < 4; ++j) v[j] = __builtin_elementwise_max(pack2(acc[q >> 2][0][4 * (q & 3) + j], acc[q >> 2][1][4 * (q & 3) + j]), 1u);
                __builtin_amdgcn_raw_buffer_store_b128(v, rs, lane * 16 + q * 1024, OFF_LP + tile * 32768u + wave * 8192, AUX_SYS);
            }
            WAIT_V(0);
            BAR();
            if (threadIdx.x == 0)
                __hip_atomic_store(ctl(me, L_Q + 2 * d) + (pos & 63), (call << 8) | tile, __ATOMIC_RELAXED, __HIP_MEMORY_SCOPE_AGENT);
            TS(7);
        }
        if (step < 2) {
#pragma unroll
            for (int m = 0; m < 2; ++m)
#pragma unroll
                for (int i = 0; i < 2; ++i) {
                    const f32x16 t = acc2[m][i];
                    acc2[m][i] = acc[m][i];
                    acc[m][i] = t;
                }
        }
    }

    if (own) {
        __amdgpu_buffer_rsrc_t rs = rsrc(me, OFF_LP);
        const int far = own - 1, tile = ro * 16 + co;
        const unsigned long long t0 = __builtin_amdgcn_s_memrealtime();
        for (unsigned rem = far ? 5 : 1; rem && !dead;) {
            const unsigned ok = (unsigned)wave_wait(ctl(me, L_FP + (far + ((lane >> 1) & far)) * 64 + tile), lane < 4, 8 * call, rem, tmo, call);
            if (!ok) { dead = true; break; }
            const int s2 = __builtin_ctz(ok & rem);
            TS(8 + 5 * far);
            const unsigned ofs = par + OFF_PS + ((far + (s2 >> 1)) * 64 + tile) * 32768u + wave * 8192;
            u32x4 pv[8];
#pragma unroll
            for (int q = 0; q < 8; ++q) pv[q] = __builtin_amdgcn_raw_buffer_load_b128(rs, lane * 16 + q * 1024, ofs, AUX_SYS);
            unsigned mn = ~0u;
#pragma unroll
            for (int q = 0; q < 8; ++q) mn = __builtin_elementwise_min(__builtin_elementwise_min(mn, __builtin_elementwise_min(pv[q][0], pv[q][1])), __builtin_elementwise_min(pv[q][2], pv[q][3]));
            if (__builtin_amdgcn_ballot_w64(mn == 0)) {
                if (__builtin_amdgcn_s_memrealtime() - t0 > DEADLINE) { __hip_atomic_store(tmo, call, __ATOMIC_RELAXED, __HIP_MEMORY_SCOPE_AGENT); dead = true; }
                continue;
            }
#pragma unroll
            for (int q = 0; q < 8; ++q) {
                __builtin_amdgcn_raw_buffer_store_b128(u32x4{0, 0, 0, 0}, rs, lane * 16 + q * 1024, ofs, AUX_SYS);
#pragma unroll
                for (int j = 0; j < 4; ++j) {
                    acc[q >> 2][0][4 * (q & 3) + j] += __builtin_bit_cast(float, pv[q][j] << 16);
                    acc[q >> 2][1][4 * (q & 3) + j] += __builtin_bit_cast(float, pv[q][j] & 0xffff0000u);
                }
            }
            rem &= ~(1u << s2);
        }
        if (dead) poison(acc);
        unsigned* op = (unsigned*)(P.out + (size_t)(128 * ro + 64 * (wave >> 1) + 4 * (lane >> 5)) * 4096
                                   + 2048 * ((rank >> 2) ^ far) + 128 * co + 64 * (wave & 1) + 2 * (lane & 31));
#pragma unroll
        for (int q = 0; q < 8; ++q)
#pragma unroll
            for (int j = 0; j < 4; j += 2) {
                const int e = 4 * (q & 3) + j;
                const f32x2 g0 = gelu2(f32x2{acc[q >> 2][0][e], acc[q >> 2][0][e + 1]}), g1 = gelu2(f32x2{acc[q >> 2][1][e], acc[q >> 2][1][e + 1]});
                op[(size_t)(32 * (q >> 2) + 8 * (q & 3) + j) * 2048] = pack2(g0[0], g1[0]);
                op[(size_t)(32 * (q >> 2) + 8 * (q & 3) + j + 1) * 2048] = pack2(g0[1], g1[1]);
            }
        TS(9);
    }
    if (threadIdx.x == 0) __hip_atomic_store(ctl(me, L_SEQ + b), call, __ATOMIC_RELAXED, __HIP_MEMORY_SCOPE_AGENT);
}

extern "C" void kernel_launch(int rank, int nranks, ncclComm_t comm,
                              hipStream_t stream,
                              void* const* d_in, const int* in_sizes, int n_in,
                              void* d_out, int out_size,
                              const int* in_dtypes, int out_dtype,
                              void* const* d_out_all,
                              void* const* d_ws_all, size_t ws_bytes) {
    if (nranks != NR || (unsigned)rank >= (unsigned)NR || n_in != 2 || in_sizes[0] != 4096 * 512 || in_sizes[1] != 512 * 4096 ||
        out_size != 512 * 4096 || ws_bytes < WS_NEED) {
        fprintf(stderr, "kernel_launch: unexpected problem shape\n");
        return;
    }
    Params P;
    P.x = (const uint16_t*)d_in[0]; P.w = (const uint16_t*)d_in[1]; P.out = (uint16_t*)d_out;
    for (int p = 0; p < NR; ++p) P.ws[p] = (char*)d_ws_all[p];
    P.rank = rank;
    fused_gemm_rs<<<256, 256, 0, stream>>>(P);
    HIP_CHECK(hipGetLastError());
}
```

```cpp
#include <hip/hip_runtime.h>
#include <rccl/rccl.h>
#include <cstdio>
#include <stdint.h>

#define HIP_CHECK(expr) do { hipError_t _e = (expr); if (_e != hipSuccess) { \
    fprintf(stderr, "HIP error %s at %s:%d\n", hipGetErrorString(_e), __FILE__, __LINE__); \
    return; } } while (0)

typedef short bf16x8 __attribute__((ext_vector_type(8)));
typedef float f32x16 __attribute__((ext_vector_type(16)));
typedef unsigned u32x4 __attribute__((ext_vector_type(4)));
typedef __bf16 bf16x2 __attribute__((ext_vector_type(2)));
typedef float f32x2 __attribute__((ext_vector_type(2)));
#define DI __device__ __forceinline__

constexpr int NR = 8;
constexpr unsigned MiB = 1u << 20;
constexpr unsigned OFF_XS = 0;
constexpr unsigned OFF_WS = 8 * MiB;
constexpr unsigned OFF_PS = 18 * MiB;
constexpr unsigned OFF_RI = 24 * MiB;
constexpr unsigned PAR = 25 * MiB;
constexpr unsigned OFF_LP = 2 * PAR;
constexpr unsigned OFF_CTL = OFF_LP + 8 * MiB;
constexpr unsigned L_CX = 0;
constexpr unsigned L_CW = L_CX + 128;
constexpr unsigned L_FP = L_CW + 160;
constexpr unsigned L_SEQ = L_FP + 192;
constexpr unsigned L_TMO = L_SEQ + 256;
constexpr unsigned L_QT = L_TMO + 1;
constexpr unsigned L_Q = L_QT + 8;
constexpr unsigned L_TK = L_Q + 16;
constexpr unsigned L_RF = L_TK + 8;
constexpr unsigned L_END = L_RF + 16;
constexpr unsigned OFF_DBG = OFF_CTL + 512 * 1024;
constexpr unsigned WS_NEED = OFF_CTL + MiB;
static_assert(L_END * 128 <= 512 * 1024, "ctl");
#define LINK_LIST 1, 2, 3, 4, 5, 6, 7, 1, 2, 3, 4, 5, 6, 7, 0, 0, 0, 0, 0, 0, 0, 0, 0, 0, 0, 0, 0, 0, 0, 0, 0, 0
constexpr unsigned char LINKS[32] = {LINK_LIST};
constexpr unsigned long long lm16(int h, int idx) {
    unsigned long long v = 0;
    for (int i = 0; i < 16; ++i) { int c = 0; for (int j = 0; j < 16 * h + i; ++j) c += LINKS[j] == LINKS[16 * h + i]; v |= (unsigned long long)(idx ? (c & 7) : LINKS[16 * h + i]) << (4 * i); }
    return v;
}
constexpr unsigned lmcnt() { unsigned v = 0; for (int i = 0; i < 32; ++i) if (LINKS[i]) v += 1u << (4 * LINKS[i]); return v; }
#define RC0 9
#define RLAG RCN
#define RCN 3
#define MAPV 1
#define OUT_AUX 16
#define PACE 113
#define PACE_SLACK 40
#define STAG0 30
constexpr unsigned relay_seq() { unsigned v = 0; int n = 0; for (int i = 0; i < RCN + RLAG; ++i) { if (i < RCN) v |= (unsigned)i << (4 * n++); if (i >= RLAG) v |= (4u + i - RLAG) << (4 * n++); } return v; }
#define HINT 4
constexpr unsigned long long DEADLINE = 100000000ull;
constexpr int AUX_SYS = 17;

constexpr int RING = 4, STAGE_B = 32768, LDS_POLL = RING * STAGE_B, LDS_BYTES = LDS_POLL + 2048;

struct Params {
    const uint16_t* x; const uint16_t* w; uint16_t* out;
    char* ws[NR];
    int rank;
};

#define TS(i) do {} while (0)
#define TSW(i) do {} while (0)

DI unsigned* ctl(char* ws, unsigned line) { return (unsigned*)(ws + OFF_CTL + line * 128u); }
DI __amdgpu_buffer_rsrc_t rsrc(const void* p, unsigned bytes) { return __builtin_amdgcn_make_buffer_rsrc((void*)p, 0, bytes, 0x00020000); }
DI int rfl(int v) { return __builtin_amdgcn_readfirstlane(v); }
#define WAIT_V(n) asm volatile("s_waitcnt vmcnt(" #n ")" ::: "memory")
#define BAR() do { asm volatile("s_waitcnt lgkmcnt(0)" ::: "memory"); __builtin_amdgcn_s_barrier(); asm volatile("" ::: "memory"); } while (0)

#define LDFL " sc0 sc1"
DI void glds4k(const char* g, unsigned lds) {
    unsigned keep;
    asm volatile("s_mov_b32 %0, m0\n\ts_mov_b32 m0, %2\n\ts_nop 0\n\t"
                 "global_load_lds_dwordx4 %1, off" LDFL "\n\t"
                 "global_load_lds_dwordx4 %1, off offset:1024" LDFL "\n\t"
                 "global_load_lds_dwordx4 %1, off offset:2048" LDFL "\n\t"
                 "global_load_lds_dwordx4 %1, off offset:3072" LDFL "\n\t"
                 "s_mov_b32 m0, %0" : "=&s"(keep) : "v"(g), "s"(lds) : "memory");
}
template <int OFS> DI void glds1k(const char* g, unsigned lds) {
    unsigned keep;
    asm volatile("s_mov_b32 %0, m0\n\ts_mov_b32 m0, %2\n\ts_nop 0\n\t"
                 "global_load_lds_dwordx4 %1, off offset:%c3" LDFL "\n\t"
                 "s_mov_b32 m0, %0" : "=&s"(keep) : "v"(g), "s"(lds), "i"(OFS) : "memory");
}
DI void glds4k_s(const char* g, unsigned v, unsigned lds) {
    unsigned keep;
    asm volatile("s_mov_b32 %0, m0\n\ts_mov_b32 m0, %3\n\ts_nop 0\n\t"
                 "global_load_lds_dwordx4 %1, %2 sc0 sc1\n\t"
                 "global_load_lds_dwordx4 %1, %2 offset:1024 sc0 sc1\n\t"
                 "global_load_lds_dwordx4 %1, %2 offset:2048 sc0 sc1\n\t"
                 "global_load_lds_dwordx4 %1, %2 offset:3072 sc0 sc1\n\t"
                 "s_mov_b32 m0, %0" : "=&s"(keep) : "v"(v), "s"(g), "s"(lds) : "memory");
}
DI void glds4k_x(const char* g, unsigned v, unsigned lds) {
    unsigned keep;
    asm volatile("s_mov_b32 %0, m0\n\ts_mov_b32 m0, %3\n\ts_nop 0\n\t"
                 "global_load_lds_dwordx4 %1, %2\n\ts_add_u32 m0, m0, 992\n\ts_nop 0\n\t"
                 "global_load_lds_dwordx4 %1, %2 offset:32\n\ts_add_u32 m0, m0, 992\n\ts_nop 0\n\t"
                 "global_load_lds_dwordx4 %1, %2 offset:64\n\ts_add_u32 m0, m0, 992\n\ts_nop 0\n\t"
                 "global_load_lds_dwordx4 %1, %2 offset:96\n\t"
                 "s_mov_b32 m0, %0" : "=&s"(keep) : "v"(v), "s"(g), "s"(lds) : "memory", "scc");
}
DI void glds_word(const unsigned* g, unsigned lds) {
    unsigned keep;
    asm volatile("s_mov_b32 %0, m0\n\ts_mov_b32 m0, %2\n\ts_nop 0\n\t"
                 "global_load_lds_dword %1, off sc0 sc1\n\t"
                 "s_mov_b32 m0, %0" : "=&s"(keep) : "v"(g), "s"(lds) : "memory");
}

DI unsigned long long wave_wait(const unsigned* p, bool active, unsigned target, unsigned long long need, unsigned* tmo, unsigned call) {
    unsigned long long t0 = __builtin_amdgcn_s_memrealtime();
    for (unsigned it = 0;; ++it) {
        unsigned v = 0;
        if (active) v = __hip_atomic_load(p, __ATOMIC_RELAXED, __HIP_MEMORY_SCOPE_SYSTEM);
        unsigned long long ok = __builtin_amdgcn_ballot_w64(active && v >= target);
        if (ok & (ok >> 1) & need) return ok;
        if ((it & 63) == 63) {
            unsigned t = rfl(__hip_atomic_load(tmo, __ATOMIC_RELAXED, __HIP_MEMORY_SCOPE_AGENT));
            if (t == call || __builtin_amdgcn_s_memrealtime() - t0 > DEADLINE) {
                __hip_atomic_store(tmo, call, __ATOMIC_RELAXED, __HIP_MEMORY_SCOPE_AGENT);
                return 0;
            }
        }
        __builtin_amdgcn_s_sleep(4);
    }
}

DI int brick_row(int r, int rb) {
    if (rb < 4) return 512 * (r ^ 1) + 128 * rb;
    if (rb < 12) return 512 * (r ^ (4 + (rb & 3))) + 256 * ((r >> 1) & 1) + 128 * ((rb - 4) >> 2);
    return 512 * r + 128 * (rb - 12);
}

#define ORDER_LIST 0, 16, 1, 17, 2, 3, 18, 19, 4, 20, 5, 21, 6, 22, 7, 23, 8, 24, 9, 25, 10, 26, 11, 27, 28, 29, 30, 31, 12, 13, 14, 15
constexpr unsigned char ORDER[32] = {ORDER_LIST};
static_assert(ORDER[0] < 4 && ORDER[1] >= 16 && ORDER[2] < 4 && ORDER[3] >= 16 && (lmcnt() >> 8) == 0x222222, "fixed start of the operand links");
constexpr unsigned long long ord8(int g) { unsigned long long v = 0; for (int i = 0; i < 8; ++i) v |= (unsigned long long)ORDER[8 * g + i] << (8 * i); return v; }
constexpr unsigned long long pos8() { unsigned long long v = 0; for (int q = 0; q < 32; ++q) if (ORDER[q] >= 4 && ORDER[q] < 12) v |= (unsigned long long)q << (8 * (ORDER[q] - 4)); return v; }
DI int order_at(int q) {
    const unsigned long long v = (q & 16) ? ((q & 8) ? ord8(3) : ord8(2)) : ((q & 8) ? ord8(1) : ord8(0));
    return (int)(v >> (8 * (q & 7))) & 31;
}
struct Piece { int kind; int seg; int blk; int sub; };
DI Piece decode_local(int n) {
    Piece p;
    int rnd = n >= 768; n -= rnd * 768;
    p.sub = n & 15;
    if (n < 512) { p.kind = 1; p.seg = (n >> 8) * 8 + ((n >> 7) & 1); p.blk = rnd * 8 + ((n >> 4) & 7); }
    else { n -= 512; p.kind = 0; p.seg = n >> 7; int q = (n >> 4) & 7; p.blk = rnd ? 4 + q : (q < 4 ? q : q + 8); }
    return p;
}
DI void piece_load(const Params& P, const Piece& p, int dst, int lane, unsigned (&r)[16]) {
    const int k0 = 256 * ((p.seg >= 2 && p.seg < 8) ? (dst & 1) : (p.seg & 1)) + 64 * (p.sub >> 2);
    if (p.kind == 0) {
        const int row = brick_row(dst, p.blk) + 32 * (p.sub & 3) + (lane & 31);
        const u32x4* s = (const u32x4*)(P.x + (size_t)row * 512 + k0 + 8 * (lane >> 5));
#pragma unroll
        for (int ks = 0; ks < 4; ++ks) { u32x4 v = s[2 * ks]; r[4 * ks] = v[0]; r[4 * ks + 1] = v[1]; r[4 * ks + 2] = v[2]; r[4 * ks + 3] = v[3]; }
    } else {
        const int col = 2048 * ((dst >> 2) ^ (p.seg >> 3)) + 128 * p.blk + 64 * ((p.sub >> 1) & 1) + 2 * (lane & 31);
        const int k = k0 + 32 * (p.sub & 1) + 8 * (lane >> 5);
        const unsigned* s = (const unsigned*)(P.w + (size_t)k * 4096 + col);
#pragma unroll
        for (int ks = 0; ks < 2; ++ks)
#pragma unroll
            for (int j = 0; j < 8; ++j) r[8 * ks + j] = s[(16 * ks + j) * 2048];
    }
}
DI int ps_index(int r, int t) {
    const int rb = t >> 4, cb = t & 15;
    return rb < 4 ? t : (1 + (r & 1)) * 64 + (2 * ((r >> 1) & 1) + ((rb - 4) >> 2)) * 16 + cb;
}
DI void piece_xform(const Piece& p, const unsigned (&r)[16], u32x4 (&o)[4]) {
    if (p.kind == 1) {
#pragma unroll
        for (int ks = 0; ks < 2; ++ks)
#pragma unroll
            for (int m = 0; m < 4; ++m) {
                o[ks][m] = __builtin_amdgcn_perm(r[8 * ks + 2 * m + 1], r[8 * ks + 2 * m], 0x05040100u);
                o[2 + ks][m] = __builtin_amdgcn_perm(r[8 * ks + 2 * m + 1], r[8 * ks + 2 * m], 0x07060302u);
            }
    } else {
#pragma unroll
        for (int ks = 0; ks < 4; ++ks) o[ks] = u32x4{r[4 * ks], r[4 * ks + 1], r[4 * ks + 2], r[4 * ks + 3]};
    }
    asm volatile("" : "+v"(o[0]), "+v"(o[1]), "+v"(o[2]), "+v"(o[3]) :: "memory");
}
DI void piece_store(const Params& P, const Piece& p, int dst, int lane, unsigned par, const u32x4 (&o)[4]) {
    __amdgpu_buffer_rsrc_t rs = rsrc(P.ws[dst & 7], OFF_LP);
    const unsigned off = par + p.sub * 4096u + (p.kind ? OFF_WS : OFF_XS) + p.seg * MiB + p.blk * 65536u;
#pragma unroll
    for (int q = 0; q < 4; ++q) __builtin_amdgcn_raw_buffer_store_b128(o[q], rs, off + lane * 16 + q * 1024, 0, AUX_SYS);
}
DI void piece_flag(const Params& P, const Piece& p, int dst, int lane) {
    const unsigned line = (p.kind ? L_CW : L_CX) + p.seg * 16 + p.blk;
    if (lane == 0) __hip_atomic_fetch_add(ctl(P.ws[dst & 7], line), 1u, __ATOMIC_RELAXED, __HIP_MEMORY_SCOPE_SYSTEM);
}

DI unsigned pack2(float a, float b) { f32x2 v = {a, b}; return __builtin_bit_cast(unsigned, __builtin_convertvector(v, bf16x2)); }
DI f32x2 gelu2(f32x2 y) {
    const float c2 = -2.0f * 0.7978845608028654f * 1.4426950408889634f;
    const f32x2 t = (y * y * (0.044715f * c2) + c2) * y;
    const f32x2 d = f32x2{__builtin_amdgcn_exp2f(t[0]), __builtin_amdgcn_exp2f(t[1])} + 1.0f;
    return y * f32x2{__builtin_amdgcn_rcpf(d[0]), __builtin_amdgcn_rcpf(d[1])};
}

DI void poison(f32x16 (&a)[2][2]) {
#pragma unroll
    for (int m = 0; m < 2; ++m)
#pragma unroll
        for (int i = 0; i < 2; ++i)
#pragma unroll
            for (int e = 0; e < 16; ++e) a[m][i][e] = __builtin_nanf("");
}

__global__ __launch_bounds__(256, 1) void fused_gemm_rs(Params P) {
    __shared__ __attribute__((aligned(1024))) char lds[LDS_BYTES];
    const int b = blockIdx.x, lane = threadIdx.x & 63, wave = rfl(threadIdx.x >> 6);
    const int xcd = b & 7, wi = b >> 3;
    const int rank = P.rank;
    char* const me = P.ws[rank & 7];
    TS(0);
    const unsigned seq0 = __hip_atomic_load(ctl(me, L_SEQ + b), __ATOMIC_RELAXED, __HIP_MEMORY_SCOPE_AGENT);
    unsigned* const tmo = ctl(me, L_TMO);
    u32x4 o[4];

    const unsigned ldsb = (unsigned)(size_t)lds;
    if (wi < 8) {
        const int i = wave * 8 + wi;
        const int d = (int)((i & 16 ? lm16(1, 0) : lm16(0, 0)) >> (4 * (i & 15))) & 7, dst = rank ^ d;
        const int nw = 8 * ((lmcnt() >> (4 * d)) & 15);
        const int nin = d == 1 ? 0 : (d < 4 ? 512 : 480);
        const int nend = d == 0 ? 0 : nin + (d == 1 ? 512 - 32 * RCN : (d < 4 ? 32 * RCN : 256));
        int sk1 = 99, sk2 = 99;
        if (d >= 4) { const int a = (pos8() >> (8 * (d - 4))) & 31, c = (pos8() >> (8 * d)) & 31; sk1 = a < c ? a : c; sk2 = a < c ? c : a; }
        const int lseg = (d >> 2) * 8 + (dst & 1);
        const unsigned vx = (lane & 31) * 1024 + (lane >> 5) * 16, vl = lane * 16;
        const __amdgpu_buffer_rsrc_t rd = rsrc(P.ws[dst & 7], OFF_LP);
        const unsigned lb = ldsb + wave * 32768;
        unsigned* const snap = (unsigned*)(lds + wave * 32768 + 16384);
        const unsigned* const rp = (d == 1 || lane < 32) ? ctl(me, L_Q + 2 * d) + lane
                                 : ctl(me, lane < 48 ? L_CW + lseg * 16 + (lane & 15) : L_RF + ((d & 1) ^ 1) * 8 + (lane & 7));
        unsigned* const tkp = ctl(me, L_TK + d);
        unsigned call = 0, par = 0, ent = 0, d2 = 0, f2 = 0, one = 1;
        int ws = 0;
        auto fetch = [&](int u, bool ld) -> bool {
            const unsigned l = rfl(lb + (ws & 3) * 4096);
            if (u >= nin) {
                int qi = ((u - nin) >> 3) & 63;
                const int sub = (u - nin) & 7;
                const bool side = (d & 6) == 2;
                if (side) qi = ((relay_seq() >> (4 * ((qi >> 1) & 7))) & 7) * 2 + (qi & 1);
                if (side && qi >= 8) {
                    const int sl = qi & 7, t = (2 * (sl & 1) + ((d & 1) ^ 1)) * 16 + RC0 + (sl >> 1);
                    if ((unsigned)__builtin_amdgcn_readlane(ent, 48 + sl) < 8 * call) return false;
                    d2 = OFF_PS + t * 32768u + sub * 4096u; f2 = L_FP + t;
                    glds4k_s(me + par + OFF_RI + (((d & 1) ^ 1) * 8 + sl) * 32768u + sub * 4096u, vl, l);
                } else {
                    const unsigned e = __builtin_amdgcn_readlane(ent, qi);
                    if (e < call << 8) return false;
                    const int t = (e & 255) < 192 ? (e & 255) : 0, pi = ps_index(rank, t), ri = (d & 1) * 8 + (qi & 7);
                    d2 = (side ? OFF_RI + ri * 32768u : OFF_PS + pi * 32768u) + sub * 4096u; f2 = side ? L_RF + ri : L_FP + pi;
                    glds4k_s(me + OFF_LP + t * 32768u + sub * 4096u, vl, l);
                }
            } else {
                int q = u >> 4; q += q >= sk1; q += q >= sk2;
                const int bk = order_at(q & 31), blk = bk & 15, sub = u & 15;
                if (bk >= 16) {
                    if (ld && (unsigned)__builtin_amdgcn_readlane(ent, 32 + blk) < 16 * call) return false;
                    d2 = OFF_WS + d * MiB + blk * 65536u + sub * 4096u; f2 = L_CW + d * 16 + blk;
                    if (ld) glds4k_s(me + par + OFF_WS + lseg * MiB + blk * 65536u + sub * 4096u, vl, l);
                } else {
                    d2 = OFF_XS + d * MiB + blk * 65536u + sub * 4096u; f2 = L_CX + d * 16 + blk;
                    if (ld) glds4k_x((const char*)P.x + (brick_row(dst, blk) + 32 * (sub & 3)) * 1024 + (dst & 1) * 512 + (sub >> 2) * 128, vx, l);
                }
            }
            ++ws;
            return true;
        };
        const int u0 = (int)(((i & 16 ? lm16(1, 1) : lm16(0, 1)) >> (4 * (i & 15))) & 7) * 8 + xcd;
        bool v0 = false, v1 = false;
        unsigned d0 = 0, f0 = 0, d1 = 0, f1 = 0, tq = 0;
        int tk = u0, tl = 0;
        auto claim = [&]() {
            asm volatile("s_mov_b32 exec_lo, %3\n\ts_mov_b32 exec_hi, 0\n\tglobal_atomic_add %0, %1, %2, off sc0 sc1\n\ts_mov_b64 exec, -1"
                         : "+v"(tq) : "v"(tkp), "v"(one), "s"(rfl(1 + tl)) : "memory");
            tl ^= 1;
        };
        const int ns = d == 1 ? 2 : 6;
        int sp = 0, rs = 0;
        unsigned pf = ~0u, cf = ~0u, idle = 0, tnext = 0;
        auto send = [&]() {
            unsigned now = (unsigned)__builtin_amdgcn_s_memrealtime();
            while ((int)(now - tnext) < 0) { __builtin_amdgcn_s_sleep(1); now = (unsigned)__builtin_amdgcn_s_memrealtime(); }
            tnext = ((int)(now - tnext) > PACE_SLACK ? now - PACE_SLACK : tnext) + PACE;
#pragma unroll
            for (int q = 0; q < 4; ++q) __builtin_amdgcn_raw_buffer_store_b128(o[q], rd, vl + par + d0 + q * 1024, 0, AUX_SYS);
        };
        snap[lane] = 0; snap[64 + lane] = 0;
        if (d > 1) {
            unsigned raw[16], raw2[16];
            const Piece p1 = {1, d, ORDER[1] & 15, u0 & 15}, p3 = {1, d, ORDER[3] & 15, u0 & 15};
            fetch(u0, true);
            d0 = d2; cf = f2;
            ws = 2; fetch(u0 + 32, true);
            v1 = true; d1 = d2; f1 = f2;
            piece_load(P, p1, dst, lane, raw);
            piece_load(P, p3, dst, lane, raw2);
            __builtin_amdgcn_sched_barrier(0);
            call = rfl(seq0) + 1; par = (call & 1) * PAR;
            WAIT_V(36);
            tnext = (unsigned)__builtin_amdgcn_s_memrealtime() + ((u0 & 7) * 2 + (u0 >> 3)) * PACE / 16 - STAG0;
#pragma unroll
            for (int q = 0; q < 4; ++q) o[q] = *(const u32x4*)(lds + wave * 32768 + q * 1024 + vl);
            send();
            __builtin_amdgcn_sched_barrier(0);
            fetch(u0 + 16, false);
            v0 = true; d0 = d2; f0 = f2; ws = 3; rs = 1; tk = u0 + 48;
            piece_xform(p1, raw, o);
#pragma unroll
            for (int q = 0; q < 4; ++q) *(u32x4*)(lds + wave * 32768 + 4096 + q * 1024 + vl) = o[q];
            piece_xform(p3, raw2, o);
#pragma unroll
            for (int q = 0; q < 4; ++q) *(u32x4*)(lds + wave * 32768 + 3 * 4096 + q * 1024 + vl) = o[q];
        } else {
            call = rfl(seq0) + 1; par = (call & 1) * PAR;
            asm volatile("" : "+s"(call), "+s"(par));
        }
        const unsigned tofs = ns * nw - (call - 1) * (nend - (ns - 2) * nw);
        const unsigned long long t0 = __builtin_amdgcn_s_memrealtime();
        if (d <= 1) tnext = (unsigned)t0;
        while (nend) {
            const bool S = v0, F = pf != ~0u;
            const unsigned sv = snap[sp * 64 + lane];
            if (S) {
#pragma unroll
                for (int q = 0; q < 4; ++q) o[q] = *(const u32x4*)(lds + wave * 32768 + (rs & 3) * 4096 + q * 1024 + vl);
                ++rs;
            }
            if (F) asm volatile("s_mov_b64 exec, 1\n\tglobal_atomic_add %0, %1, off sc1\n\ts_mov_b64 exec, -1" :: "v"(ctl(P.ws[dst & 7], pf)), "v"(one) : "memory");
            if (!S && !v1 && cf == ~0u && tk >= nend) break;
            ent = sv > ent ? sv : ent;
            if (S) send();
            const bool E = S && f0 - L_FP < 192u;
            if (E) asm volatile("s_mov_b64 exec, 1\n\tglobal_atomic_add %0, %1, off sc1\n\ts_mov_b64 exec, -1" :: "v"(ctl(P.ws[dst & 7], f0)), "v"(one) : "memory");
            asm volatile("" : "+v"(ent) :: "memory");
            const bool ld = ws >= ns - 2, L = tk < nend && fetch(tk, ld);
            if (L && ld) claim();
            glds_word(rp, rfl(lb + 16384 + sp * 256));
            if (!S) WAIT_V(0);
            else if (L && ld) { if (F && E) WAIT_V(12); else if (F || E) WAIT_V(11); else WAIT_V(10); }
            else { if (F && E) WAIT_V(7); else if (F || E) WAIT_V(6); else WAIT_V(5); }
            asm volatile("" : "+v"(tq) :: "memory");
            pf = cf; cf = S && !E ? f0 : ~0u;
            v0 = v1; d0 = d1; f0 = f1;
            v1 = L; d1 = d2; f1 = f2;
            if (S) sp ^= 1;
            else if (!v0) { v0 = v1; d0 = d1; f0 = f1; v1 = false; }
            if (L) tk = ws < ns ? u0 + ws * nw : (int)(__builtin_amdgcn_readlane(tq, rfl(tl)) + tofs);
            else if (!S && (++idle & 63) == 0) {
                const unsigned t = rfl(__hip_atomic_load(tmo, __ATOMIC_RELAXED, __HIP_MEMORY_SCOPE_AGENT));
                if (t == call || __builtin_amdgcn_s_memrealtime() - t0 > DEADLINE) {
                    __hip_atomic_store(tmo, call, __ATOMIC_RELAXED, __HIP_MEMORY_SCOPE_AGENT);
                    break;
                }
            }
        }
        TSW(3);
        if (threadIdx.x == 0) __hip_atomic_store(ctl(me, L_SEQ + b), call, __ATOMIC_RELAXED, __HIP_MEMORY_SCOPE_AGENT);
        return;
    }

    const int grp = (wi - 8) >> 2, own = grp >= 4 ? 0 : 1 + (grp & 1);
    const int r4 = xcd & 3, cb = 8 * (grp & 1) + 2 * (wi & 3) + (xcd >> 2);
    const int ro = 2 * (xcd & 1) + ((grp >> 1) & 1), co = 4 * (xcd & 2) + 2 * (wi & 3) + (xcd >> 2);
    for (int i = threadIdx.x; i < 512; i += 256) ((unsigned*)(lds + LDS_POLL))[i] = 0;
    bool dead = false;
    unsigned call, par;
    {
        const int n = (b - 64) * 4 + wave;
        Piece p0 = decode_local(n), p1 = decode_local(n + 768);
        auto stage = [&](int kind) {
            unsigned raw[16], raw2[16];
            p0.kind = p1.kind = kind;
            piece_load(P, p0, rank, lane, raw);
            piece_load(P, p1, rank, lane, raw2);
            __builtin_amdgcn_sched_barrier(0);
            call = rfl(seq0) + 1; par = (call & 1) * PAR;
            piece_xform(p0, raw, o);
            piece_store(P, p0, rank, lane, par, o);
            piece_xform(p1, raw2, o);
            piece_store(P, p1, rank, lane, par, o);
        };
        if (n < 512) stage(1); else stage(0);
        WAIT_V(4);
        piece_flag(P, p0, rank, lane);
        WAIT_V(0);
        piece_flag(P, p1, rank, lane);
        TS(1);
    }

    f32x16 acc[2][2], acc2[2][2];
#pragma unroll
    for (int m = 0; m < 2; ++m)
#pragma unroll
        for (int i = 0; i < 2; ++i)
#pragma unroll
            for (int e = 0; e < 16; ++e) { acc[m][i][e] = 0.f; acc2[m][i][e] = 0.f; }

    const int step0 = own == 2 ? 0 : 1, step1 = own == 1 ? 3 : 2;
    int pslot = 0;
    for (int step = step0; step < step1; ++step) {
        unsigned rem = 0x5555; int wofs = 0, rb = 12 + ro;
        const int cc = step == 1 ? cb : co;
        if (step == 0) { rem = 5; wofs = 8; }
        else if (step == 1) {
            rb = 4 * (grp >> 1) + r4;
            if (grp >= 2) rem &= ~(0x100u << (2 * r4));
        }
        const unsigned* cptr = ctl(me, (lane & 1) ? L_CW + ((((lane >> 1) & 7) + wofs) % 10) * 16 + cc : L_CX + ((lane >> 1) & 7) * 16 + rb);
        const unsigned target = 16 * call;
        const char* gA = me + par + OFF_XS + rb * 65536u + wave * 4096 + lane * 16;
        const char* gB = me + par + OFF_WS + wofs * MiB + cc * 65536u + wave * 4096 + lane * 16;
        const unsigned lw = ldsb + wave * 4096;
        const char* la = lds + (wave >> 1) * 8192 + lane * 16;
        const char* lb = lds + 16384 + (wave & 1) * 8192 + lane * 16;
        unsigned known = 0;
        bool polled = false;
        auto next = [&]() -> int {
            if (polled) {
                unsigned v = ((const unsigned*)(lds + LDS_POLL + pslot * 64))[lane & 15];
                known |= (unsigned)__builtin_amdgcn_ballot_w64(v >= target) & 0xffffu;
                ++pslot; polled = false;
            }
            if (!(known & (known >> 1) & rem)) {
                unsigned* mw = (unsigned*)(lds + LDS_POLL + pslot * 64);
                if (wave == 0) {
                    unsigned ok = (unsigned)wave_wait(cptr, lane < 16, target, rem, tmo, call);
                    if (lane == 0) *mw = ok ? ok : 0x80000000u;
                }
                BAR();
                unsigned ok = rfl(*mw);
                if (ok & 0x80000000u) { dead = true; ok = 0xffffu; }
                known |= ok;
                ++pslot;
            }
            const int s2 = __builtin_ctz(known & (known >> 1) & rem);
            rem &= ~(1u << s2);
            return s2 >> 1;
        };
        bf16x8 fa[2][4], fb[2][4];
#define SB() __builtin_amdgcn_sched_barrier(0)
#define RD(h, slot) do { _Pragma("unroll") for (int k2 = 0; k2 < 2; ++k2) { \
            fa[h][2 * k2] = *(const bf16x8*)(la + (slot) * STAGE_B + (2 * h + k2) * 1024); \
            fb[h][2 * k2] = *(const bf16x8*)(lb + (slot) * STAGE_B + h * 4096 + k2 * 1024); \
            fb[h][2 * k2 + 1] = *(const bf16x8*)(lb + (slot) * STAGE_B + h * 4096 + 2048 + k2 * 1024); \
            fa[h][2 * k2 + 1] = *(const bf16x8*)(la + (slot) * STAGE_B + 4096 + (2 * h + k2) * 1024); } SB(); } while (0)
#define MM(h, k2, m) do { acc[m][0] = __builtin_amdgcn_mfma_f32_32x32x16_bf16(fa[h][2 * k2 + m], fb[h][2 * k2], acc[m][0], 0, 0, 0); \
            acc[m][1] = __builtin_amdgcn_mfma_f32_32x32x16_bf16(fa[h][2 * k2 + m], fb[h][2 * k2 + 1], acc[m][1], 0, 0, 0); SB(); } while (0)
#define MMA(h) do { MM(h, 0, 0); MM(h, 0, 1); MM(h, 1, 0); MM(h, 1, 1); } while (0)
#define MMA_LD(h, g, l) do { const char* g_ = (g); const unsigned l_ = (l); \
            MM(h, 0, 0); glds1k<0>(g_, l_); SB(); MM(h, 0, 1); glds1k<1024>(g_, l_); SB(); \
            MM(h, 1, 0); glds1k<2048>(g_, l_); SB(); MM(h, 1, 1); glds1k<3072>(g_, l_); SB(); } while (0)
        BAR();
        unsigned so = next() * MiB;
#pragma unroll
        for (int kc = 0; kc < 3; ++kc) { glds4k(gA + so + kc * 16384, lw + kc * STAGE_B); glds4k(gB + so + kc * 16384, lw + kc * STAGE_B + 16384); }
        glds4k(gA + so + 3 * 16384, lw + 3 * STAGE_B);
        WAIT_V(20); BAR();
        RD(0, 0);
        while (rem) {
            RD(1, 0); MMA_LD(0, gB + so + 3 * 16384, lw + 3 * STAGE_B + 16384);
            WAIT_V(16); BAR();
            RD(0, 1);
            const unsigned no = next() * MiB;
            SB(); MMA_LD(1, gA + no, lw);
            RD(1, 1); MMA_LD(0, gB + no, lw + 16384);
            WAIT_V(16); BAR();
            RD(0, 2); MMA_LD(1, gA + no + 16384, lw + STAGE_B);
            RD(1, 2);
            if (rem && !(known & (known >> 1) & rem)) {
                const unsigned pa = rfl(ldsb + LDS_POLL + pslot * 64);
                if (wave == 0 && lane < 16) glds_word(cptr, pa);
                polled = true;
            }
            SB();
            MMA_LD(0, gB + no + 16384, lw + STAGE_B + 16384);
            WAIT_V(16); BAR();
            RD(0, 3); MMA_LD(1, gA + no + 2 * 16384, lw + 2 * STAGE_B);
            RD(1, 3); MMA_LD(0, gB + no + 2 * 16384, lw + 2 * STAGE_B + 16384);
            WAIT_V(16); BAR();
            RD(0, 0); MMA_LD(1, gA + no + 3 * 16384, lw + 3 * STAGE_B);
            so = no;
        }
        RD(1, 0); MMA_LD(0, gB + so + 3 * 16384, lw + 3 * STAGE_B + 16384);
        WAIT_V(16); BAR();
        RD(0, 1); MMA(1);
        RD(1, 1); MMA(0);
        WAIT_V(8); BAR();
        RD(0, 2); MMA(1);
        RD(1, 2); MMA(0);
        WAIT_V(0); BAR();
        RD(0, 3); MMA(1);
        RD(1, 3); MMA(0);
        MMA(1);
        TS(4 + step);
        if (step == 1) {
            const bool relay = grp < 2 && (unsigned)(cb - RC0) < RCN;
            const int d = relay ? 2 + (r4 & 1) : (grp < 2 ? 1 : 4 + r4), tile = rb * 16 + cb;
            unsigned pos = (cb - RC0) * 2 + (r4 >> 1);
            if (threadIdx.x == 0 && !relay)
                pos = __hip_atomic_fetch_add(ctl(me, L_QT + d), 1u, __ATOMIC_RELAXED, __HIP_MEMORY_SCOPE_AGENT) - (call - 1) * (grp < 2 ? 64 - 4 * RCN : 32);
            if (dead) poison(acc);
            __amdgpu_buffer_rsrc_t rs = rsrc(me, OFF_CTL);
#pragma unroll
            for (int q = 0; q < 8; ++q) {
                u32x4 v;
#pragma unroll
                for (int j = 0; j < 4; ++j) v[j] = __builtin_elementwise_max(pack2(acc[q >> 2][0][4 * (q & 3) + j], acc[q >> 2][1][4 * (q & 3) + j]), 1u);
                __builtin_amdgcn_raw_buffer_store_b128(v, rs, OFF_LP + tile * 32768u + wave * 8192 + lane * 16 + q * 1024, 0, AUX_SYS);
            }
            WAIT_V(0);
            BAR();
            if (threadIdx.x == 0)
                __hip_atomic_store(ctl(me, L_Q + 2 * d) + (pos & 63), (call << 8) | tile, __ATOMIC_RELAXED, __HIP_MEMORY_SCOPE_AGENT);
            TS(7);
        }
        if (step < 2) {
#pragma unroll
            for (int m = 0; m < 2; ++m)
#pragma unroll
                for (int i = 0; i < 2; ++i) {
                    const f32x16 t = acc2[m][i];
                    acc2[m][i] = acc[m][i];
                    acc[m][i] = t;
                }
        }
    }

    if (own) {
        __amdgpu_buffer_rsrc_t rs = rsrc(me, OFF_LP);
        const int far = own - 1, tile = ro * 16 + co;
        const unsigned long long t0 = __builtin_amdgcn_s_memrealtime();
        for (unsigned rem = far ? 5 : 1; rem && !dead;) {
            const unsigned ok = (unsigned)wave_wait(ctl(me, L_FP + (far + ((lane >> 1) & far)) * 64 + tile), lane < 4, 8 * call - 8 + HINT, rem, tmo, call);
            if (!ok) { dead = true; break; }
            const int s2 = __builtin_ctz(ok & rem);
            TS(8 + 5 * far);
            const unsigned ofs = par + OFF_PS + ((far + (s2 >> 1)) * 64 + tile) * 32768u + wave * 8192;
            u32x4 pv[8];
#pragma unroll
            for (int q = 0; q < 8; ++q) pv[q] = __builtin_amdgcn_raw_buffer_load_b128(rs, lane * 16 + q * 1024, ofs, AUX_SYS);
            unsigned mn = ~0u;
#pragma unroll
            for (int q = 0; q < 8; ++q) mn = __builtin_elementwise_min(__builtin_elementwise_min(mn, __builtin_elementwise_min(pv[q][0], pv[q][1])), __builtin_elementwise_min(pv[q][2], pv[q][3]));
            if (__builtin_amdgcn_ballot_w64(mn == 0)) {
                if (__builtin_amdgcn_s_memrealtime() - t0 > DEADLINE) { __hip_atomic_store(tmo, call, __ATOMIC_RELAXED, __HIP_MEMORY_SCOPE_AGENT); dead = true; }
                continue;
            }
#pragma unroll
            for (int q = 0; q < 8; ++q) {
                __builtin_amdgcn_raw_buffer_store_b128(u32x4{0, 0, 0, 0}, rs, ofs + lane * 16 + q * 1024, 0, AUX_SYS);
#pragma unroll
                for (int j = 0; j < 4; ++j) {
                    acc[q >> 2][0][4 * (q & 3) + j] += __builtin_bit_cast(float, pv[q][j] << 16);
                    acc[q >> 2][1][4 * (q & 3) + j] += __builtin_bit_cast(float, pv[q][j] & 0xffff0000u);
                }
            }
            rem &= ~(1u << s2);
        }
        if (dead) poison(acc);
        const __amdgpu_buffer_rsrc_t ors = rsrc(P.out, 512 * 4096 * 2);
        const unsigned oo = (128 * ro + 64 * (wave >> 1) + (lane >> 3)) * 8192 + 4096 * ((rank >> 2) ^ far) + 256 * co + 128 * (wave & 1) + (lane & 7) * 16;
        unsigned* const lt = (unsigned*)(lds + wave * 8192) + (lane >> 5) * 128 + (lane & 31);
#pragma unroll
        for (int q = 0; q < 8; ++q) {
#pragma unroll
            for (int j = 0; j < 4; j += 2) {
                const int e = 4 * (q & 3) + j;
                const f32x2 g0 = gelu2(f32x2{acc[q >> 2][0][e], acc[q >> 2][0][e + 1]}), g1 = gelu2(f32x2{acc[q >> 2][1][e], acc[q >> 2][1][e + 1]});
                lt[q * 256 + j * 32] = pack2(g0[0], g1[0]);
                lt[q * 256 + j * 32 + 32] = pack2(g0[1], g1[1]);
            }
            __builtin_amdgcn_fence(__ATOMIC_ACQ_REL, "wavefront");
            const u32x4 v = *(const u32x4*)(lds + wave * 8192 + q * 1024 + lane * 16);
            __builtin_amdgcn_raw_buffer_store_b128(v, ors, oo + q * 65536, 0, OUT_AUX);
        }
        TS(9);
    }
    if (threadIdx.x == 0) __hip_atomic_store(ctl(me, L_SEQ + b), call, __ATOMIC_RELAXED, __HIP_MEMORY_SCOPE_AGENT);
}

extern "C" void kernel_launch(int rank, int nranks, ncclComm_t comm,
                              hipStream_t stream,
                              void* const* d_in, const int* in_sizes, int n_in,
                              void* d_out, int out_size,
                              const int* in_dtypes, int out_dtype,
                              void* const* d_out_all,
                              void* const* d_ws_all, size_t ws_bytes) {
    if (nranks != NR || (unsigned)rank >= (unsigned)NR || n_in != 2 || in_sizes[0] != 4096 * 512 || in_sizes[1] != 512 * 4096 ||
        out_size != 512 * 4096 || ws_bytes < WS_NEED) {
        fprintf(stderr, "kernel_launch: unexpected problem shape\n");
        return;
    }
    Params P;
    P.x = (const uint16_t*)d_in[0]; P.w = (const uint16_t*)d_in[1]; P.out = (uint16_t*)d_out;
    for (int p = 0; p < NR; ++p) P.ws[p] = (char*)d_ws_all[p];
    P.rank = rank;
    fused_gemm_rs<<<256, 256, 0, stream>>>(P);
    HIP_CHECK(hipGetLastError());
}
```

```cpp
#include <hip/hip_runtime.h>
#include <rccl/rccl.h>
#include <cstdio>
#include <stdint.h>

#define HIP_CHECK(expr) do { hipError_t _e = (expr); if (_e != hipSuccess) { \
    fprintf(stderr, "HIP error %s at %s:%d\n", hipGetErrorString(_e), __FILE__, __LINE__); \
    return; } } while (0)

typedef short bf16x8 __attribute__((ext_vector_type(8)));
typedef float f32x16 __attribute__((ext_vector_type(16)));
typedef unsigned u32x4 __attribute__((ext_vector_type(4)));
typedef __bf16 bf16x2 __attribute__((ext_vector_type(2)));
typedef float f32x2 __attribute__((ext_vector_type(2)));
#define DI __device__ __forceinline__

constexpr int NR = 8;
constexpr unsigned MiB = 1u << 20;
constexpr unsigned OFF_XS = 0;
constexpr unsigned OFF_WS = 8 * MiB;
constexpr unsigned OFF_PS = 18 * MiB;
constexpr unsigned OFF_RI = 24 * MiB;
constexpr unsigned PAR = 25 * MiB;
constexpr unsigned OFF_LP = 2 * PAR;
constexpr unsigned OFF_CTL = OFF_LP + 8 * MiB;
constexpr unsigned L_CX = 0;
constexpr unsigned L_CW = L_CX + 128;
constexpr unsigned L_FP = L_CW + 160;
constexpr unsigned L_SEQ = L_FP + 192;
constexpr unsigned L_TMO = L_SEQ + 256;
constexpr unsigned L_QT = L_TMO + 1;
constexpr unsigned L_Q = L_QT + 8;
constexpr unsigned L_TK = L_Q + 16;
constexpr unsigned L_RF = L_TK + 8;
constexpr unsigned L_END = L_RF + 16;
constexpr unsigned OFF_DBG = OFF_CTL + 512 * 1024;
constexpr unsigned WS_NEED = OFF_CTL + MiB;
static_assert(L_END * 128 <= 512 * 1024, "ctl");
#define LINK_LIST 1, 2, 3, 4, 5, 6, 7, 1, 2, 3, 4, 5, 6, 7, 0, 0, 0, 0, 0, 0, 0, 0, 0, 0, 0, 0, 0, 0, 0, 0, 0, 0
constexpr unsigned char LINKS[32] = {LINK_LIST};
constexpr unsigned long long lm16(int h, int idx) {
    unsigned long long v = 0;
    for (int i = 0; i < 16; ++i) { int c = 0; for (int j = 0; j < 16 * h + i; ++j) c += LINKS[j] == LINKS[16 * h + i]; v |= (unsigned long long)(idx ? (c & 7) : LINKS[16 * h + i]) << (4 * i); }
    return v;
}
constexpr unsigned lmcnt() { unsigned v = 0; for (int i = 0; i < 32; ++i) if (LINKS[i]) v += 1u << (4 * LINKS[i]); return v; }
#define RC0 9
#define RLAG RCN
#define RCN 3
#define MAPV 2
#define OUT_AUX 16
#define PACE 113
#define PACE_SLACK 40
#define STAG0 30
constexpr unsigned relay_seq() { unsigned v = 0; int n = 0; for (int i = 0; i < RCN + RLAG; ++i) { if (i < RCN) v |= (unsigned)i << (4 * n++); if (i >= RLAG) v |= (4u + i - RLAG) << (4 * n++); } return v; }
#define HINT 4
#define POLL_SLEEP 1
#define ROT_N 5
#define ROT_O 1
#define LP_AUX 16
constexpr unsigned long long DEADLINE = 100000000ull;
constexpr int AUX_SYS = 17;

constexpr int RING = 4, STAGE_B = 32768, LDS_POLL = RING * STAGE_B, LDS_BYTES = LDS_POLL + 2048 + 256;

struct Params {
    const uint16_t* x; const uint16_t* w; char* me;
    int rank, pad;
    uint16_t* out;
    char* ws[NR];
};

#define TS(i) do {} while (0)
#define TSW(i) do {} while (0)

DI unsigned* ctl(char* ws, unsigned line) { return (unsigned*)(ws + OFF_CTL + line * 128u); }
DI __amdgpu_buffer_rsrc_t rsrc(const void* p, unsigned bytes) { return __builtin_amdgcn_make_buffer_rsrc((void*)p, 0, bytes, 0x00020000); }
DI int rfl(int v) { return __builtin_amdgcn_readfirstlane(v); }
#define WAIT_V(n) asm volatile("s_waitcnt vmcnt(" #n ")" ::: "memory")
#define BAR() do { asm volatile("s_waitcnt lgkmcnt(0)" ::: "memory"); __builtin_amdgcn_s_barrier(); asm volatile("" ::: "memory"); } while (0)

#define LDFL " sc0 sc1"
DI void glds4k(const char* g, unsigned lds) {
    unsigned keep;
    asm volatile("s_mov_b32 %0, m0\n\ts_mov_b32 m0, %2\n\ts_nop 0\n\t"
                 "global_load_lds_dwordx4 %1, off" LDFL "\n\t"
                 "global_load_lds_dwordx4 %1, off offset:1024" LDFL "\n\t"
                 "global_load_lds_dwordx4 %1, off offset:2048" LDFL "\n\t"
                 "global_load_lds_dwordx4 %1, off offset:3072" LDFL "\n\t"
                 "s_mov_b32 m0, %0" : "=&s"(keep) : "v"(g), "s"(lds) : "memory");
}
template <int OFS> DI void glds1k(const char* g, unsigned lds) {
    unsigned keep;
    asm volatile("s_mov_b32 %0, m0\n\ts_mov_b32 m0, %2\n\ts_nop 0\n\t"
                 "global_load_lds_dwordx4 %1, off offset:%c3" LDFL "\n\t"
                 "s_mov_b32 m0, %0" : "=&s"(keep) : "v"(g), "s"(lds), "i"(OFS) : "memory");
}
DI void glds4k_s(const char* g, unsigned v, unsigned lds) {
    unsigned keep;
    asm volatile("s_mov_b32 %0, m0\n\ts_mov_b32 m0, %3\n\ts_nop 0\n\t"
                 "global_load_lds_dwordx4 %1, %2 sc0 sc1\n\t"
                 "global_load_lds_dwordx4 %1, %2 offset:1024 sc0 sc1\n\t"
                 "global_load_lds_dwordx4 %1, %2 offset:2048 sc0 sc1\n\t"
                 "global_load_lds_dwordx4 %1, %2 offset:3072 sc0 sc1\n\t"
                 "s_mov_b32 m0, %0" : "=&s"(keep) : "v"(v), "s"(g), "s"(lds) : "memory");
}
DI void glds4k_x(const char* g, unsigned v, unsigned lds) {
    unsigned keep;
    asm volatile("s_mov_b32 %0, m0\n\ts_mov_b32 m0, %3\n\ts_nop 0\n\t"
                 "global_load_lds_dwordx4 %1, %2\n\ts_add_u32 m0, m0, 992\n\ts_nop 0\n\t"
                 "global_load_lds_dwordx4 %1, %2 offset:32\n\ts_add_u32 m0, m0, 992\n\ts_nop 0\n\t"
                 "global_load_lds_dwordx4 %1, %2 offset:64\n\ts_add_u32 m0, m0, 992\n\ts_nop 0\n\t"
                 "global_load_lds_dwordx4 %1, %2 offset:96\n\t"
                 "s_mov_b32 m0, %0" : "=&s"(keep) : "v"(v), "s"(g), "s"(lds) : "memory", "scc");
}
DI void glds_word(const unsigned* g, unsigned lds) {
    unsigned keep;
    asm volatile("s_mov_b32 %0, m0\n\ts_mov_b32 m0, %2\n\ts_nop 0\n\t"
                 "global_load_lds_dword %1, off sc0 sc1\n\t"
                 "s_mov_b32 m0, %0" : "=&s"(keep) : "v"(g), "s"(lds) : "memory");
}

DI unsigned long long wave_wait(const unsigned* p, bool active, unsigned target, unsigned long long need, unsigned* tmo, unsigned call) {
    unsigned long long t0 = __builtin_amdgcn_s_memrealtime();
    for (unsigned it = 0;; ++it) {
        unsigned v = 0;
        if (active) v = __hip_atomic_load(p, __ATOMIC_RELAXED, __HIP_MEMORY_SCOPE_SYSTEM);
        unsigned long long ok = __builtin_amdgcn_ballot_w64(active && v >= target);
        if (ok & (ok >> 1) & need) return ok;
        if ((it & 63) == 63) {
            unsigned t = rfl(__hip_atomic_load(tmo, __ATOMIC_RELAXED, __HIP_MEMORY_SCOPE_AGENT));
            if (t == call || __builtin_amdgcn_s_memrealtime() - t0 > DEADLINE) {
                __hip_atomic_store(tmo, call, __ATOMIC_RELAXED, __HIP_MEMORY_SCOPE_AGENT);
                return 0;
            }
        }
        __builtin_amdgcn_s_sleep(POLL_SLEEP);
    }
}

DI int brick_row(int r, int rb) {
    if (rb < 4) return 512 * (r ^ 1) + 128 * rb;
    if (rb < 12) return 512 * (r ^ (4 + (rb & 3))) + 256 * ((r >> 1) & 1) + 128 * ((rb - 4) >> 2);
    return 512 * r + 128 * (rb - 12);
}

#define ORDER_LIST 0, 16, 1, 17, 2, 3, 18, 19, 4, 20, 5, 21, 6, 22, 7, 23, 8, 24, 9, 25, 10, 26, 11, 27, 28, 29, 30, 31, 12, 13, 14, 15
constexpr unsigned char ORDER[32] = {ORDER_LIST};
static_assert(ORDER[0] < 4 && ORDER[1] >= 16 && ORDER[2] < 4 && ORDER[3] >= 16 && (lmcnt() >> 8) == 0x222222, "fixed start of the operand links");
constexpr bool rot_ok() { for (int q = 28; q < 32; ++q) if (ORDER[q] < 12) return false; return true; }
static_assert(rot_ok(), "skipped blocks stay outside the last range");
constexpr unsigned long long ord8(int g) { unsigned long long v = 0; for (int i = 0; i < 8; ++i) v |= (unsigned long long)ORDER[8 * g + i] << (8 * i); return v; }
constexpr unsigned long long pos8() { unsigned long long v = 0; for (int q = 0; q < 32; ++q) if (ORDER[q] >= 4 && ORDER[q] < 12) v |= (unsigned long long)q << (8 * (ORDER[q] - 4)); return v; }
DI int order_at(int q) {
    const unsigned long long v = (q & 16) ? ((q & 8) ? ord8(3) : ord8(2)) : ((q & 8) ? ord8(1) : ord8(0));
    return (int)(v >> (8 * (q & 7))) & 31;
}
struct Piece { int kind; int seg; int blk; int sub; };
DI Piece decode_local(int n) {
    Piece p;
    int rnd = n >= 768; n -= rnd * 768;
    p.sub = n & 15;
    if (n < 512) { p.kind = 1; p.seg = (n >> 8) * 8 + ((n >> 7) & 1); p.blk = rnd * 8 + ((n >> 4) & 7); }
    else { n -= 512; p.kind = 0; p.seg = n >> 7; int q = (n >> 4) & 7; p.blk = rnd ? 4 + q : (q < 4 ? q : q + 8); }
    return p;
}
DI void piece_load(const Params& P, const Piece& p, int dst, int lane, unsigned (&r)[16]) {
    const int k0 = 256 * ((p.seg >= 2 && p.seg < 8) ? (dst & 1) : (p.seg & 1)) + 64 * (p.sub >> 2);
    if (p.kind == 0) {
        const int row = brick_row(dst, p.blk) + 32 * (p.sub & 3) + (lane & 31);
        const u32x4* s = (const u32x4*)(P.x + (size_t)row * 512 + k0 + 8 * (lane >> 5));
#pragma unroll
        for (int ks = 0; ks < 4; ++ks) { u32x4 v = s[2 * ks]; r[4 * ks] = v[0]; r[4 * ks + 1] = v[1]; r[4 * ks + 2] = v[2]; r[4 * ks + 3] = v[3]; }
    } else {
        const int col = 2048 * ((dst >> 2) ^ (p.seg >> 3)) + 128 * p.blk + 64 * ((p.sub >> 1) & 1) + 2 * (lane & 31);
        const int k = k0 + 32 * (p.sub & 1) + 8 * (lane >> 5);
        const unsigned* s = (const unsigned*)(P.w + (size_t)k * 4096 + col);
#pragma unroll
        for (int ks = 0; ks < 2; ++ks)
#pragma unroll
            for (int j = 0; j < 8; ++j) r[8 * ks + j] = s[(16 * ks + j) * 2048];
    }
}
DI int ps_index(int r, int t) {
    const int rb = t >> 4, cb = t & 15;
    return rb < 4 ? t : (1 + (r & 1)) * 64 + (2 * ((r >> 1) & 1) + ((rb - 4) >> 2)) * 16 + cb;
}
DI void piece_xform(const Piece& p, const unsigned (&r)[16], u32x4 (&o)[4]) {
    if (p.kind == 1) {
#pragma unroll
        for (int ks = 0; ks < 2; ++ks)
#pragma unroll
            for (int m = 0; m < 4; ++m) {
                o[ks][m] = __builtin_amdgcn_perm(r[8 * ks + 2 * m + 1], r[8 * ks + 2 * m], 0x05040100u);
                o[2 + ks][m] = __builtin_amdgcn_perm(r[8 * ks + 2 * m + 1], r[8 * ks + 2 * m], 0x07060302u);
            }
    } else {
#pragma unroll
        for (int ks = 0; ks < 4; ++ks) o[ks] = u32x4{r[4 * ks], r[4 * ks + 1], r[4 * ks + 2], r[4 * ks + 3]};
    }
    asm volatile("" : "+v"(o[0]), "+v"(o[1]), "+v"(o[2]), "+v"(o[3]) :: "memory");
}
DI void piece_store(const Params& P, const Piece& p, int lane, unsigned par, const u32x4 (&o)[4]) {
    __amdgpu_buffer_rsrc_t rs = rsrc(P.me, OFF_LP);
    const unsigned off = par + p.sub * 4096u + (p.kind ? OFF_WS : OFF_XS) + p.seg * MiB + p.blk * 65536u;
#pragma unroll
    for (int q = 0; q < 4; ++q) __builtin_amdgcn_raw_buffer_store_b128(o[q], rs, off + lane * 16 + q * 1024, 0, AUX_SYS);
}
DI void piece_flag(const Params& P, const Piece& p, int lane) {
    const unsigned line = (p.kind ? L_CW : L_CX) + p.seg * 16 + p.blk;
    if (lane == 0) __hip_atomic_fetch_add(ctl(P.me, line), 1u, __ATOMIC_RELAXED, __HIP_MEMORY_SCOPE_SYSTEM);
}

DI unsigned pack2(float a, float b) { f32x2 v = {a, b}; return __builtin_bit_cast(unsigned, __builtin_convertvector(v, bf16x2)); }
DI f32x2 gelu2(f32x2 y) {
    const float c2 = -2.0f * 0.7978845608028654f * 1.4426950408889634f;
    const f32x2 t = (y * y * (0.044715f * c2) + c2) * y;
    const f32x2 d = f32x2{__builtin_amdgcn_exp2f(t[0]), __builtin_amdgcn_exp2f(t[1])} + 1.0f;
    return y * f32x2{__builtin_amdgcn_rcpf(d[0]), __builtin_amdgcn_rcpf(d[1])};
}

DI void poison(f32x16 (&a)[2][2]) {
#pragma unroll
    for (int m = 0; m < 2; ++m)
#pragma unroll
        for (int i = 0; i < 2; ++i)
#pragma unroll
            for (int e = 0; e < 16; ++e) a[m][i][e] = __builtin_nanf("");
}

__global__ __launch_bounds__(256, 1) void fused_gemm_rs(Params P) {
    __shared__ __attribute__((aligned(1024))) char lds[LDS_BYTES];
    const int b = blockIdx.x, lane = threadIdx.x & 63, wave = rfl(threadIdx.x >> 6);
    const int xcd = b & 7, wi = b >> 3;
    const int rank = P.rank;
    char* const me = P.me;
    TS(0);
    const unsigned seq0 = __hip_atomic_load(ctl(me, L_SEQ + b), __ATOMIC_RELAXED, __HIP_MEMORY_SCOPE_AGENT);
    unsigned* const tmo = ctl(me, L_TMO);
    u32x4 o[4];

    const unsigned ldsb = (unsigned)(size_t)lds;
    if (wi < 8) {
        const int i = wave * 8 + wi;
        const int d = (int)((i & 16 ? lm16(1, 0) : lm16(0, 0)) >> (4 * (i & 15))) & 7, dst = rank ^ d;
        const int u0 = (int)(((i & 16 ? lm16(1, 1) : lm16(0, 1)) >> (4 * (i & 15))) & 7) * 8 + xcd;
        const unsigned vx = (lane & 31) * 1024 + (lane >> 5) * 16, vl = lane * 16;
        const unsigned lb = ldsb + wave * 32768;
        if (d > 1) {
            const char* xs = (const char*)P.x + (512 * (dst ^ 1) + 32 * (u0 & 3)) * 1024 + (dst & 1) * 512 + ((u0 >> 2) & 3) * 128;
            glds4k_x(xs + ORDER[0] * 131072, vx, lb);
            glds4k_x(xs + ORDER[2] * 131072, vx, lb + 8192);
        }
        __builtin_amdgcn_sched_barrier(0);
        const int nw = 8 * ((lmcnt() >> (4 * d)) & 15);
        const int nin = d == 1 ? 0 : (d < 4 ? 512 : 480);
        const int nend = d == 0 ? 0 : nin + (d == 1 ? 512 - 32 * RCN : (d < 4 ? 32 * RCN : 256));
        int sk1 = 99, sk2 = 99;
        const int rot = (d < 2 ? 0 : 7 - d) * ROT_N / 6;
        auto unrot = [&](int a) { return a < 28 - ROT_N ? a : (a - rot < 28 - ROT_N ? a - rot + ROT_N : a - rot); };
        if (d >= 4) { const int a = unrot((pos8() >> (8 * (d - 4))) & 31), c = unrot((pos8() >> (8 * d)) & 31); sk1 = a < c ? a : c; sk2 = a < c ? c : a; }
        const int lseg = (d >> 2) * 8 + (dst & 1);
        const __amdgpu_buffer_rsrc_t rd = rsrc(P.ws[dst & 7], OFF_LP);
        unsigned* const snap = (unsigned*)(lds + wave * 32768 + 16384);
        const unsigned* const rp = (d == 1 || lane < 32) ? ctl(me, L_Q + 2 * d) + lane
                                 : ctl(me, lane < 48 ? L_CW + lseg * 16 + (lane & 15) : L_RF + ((d & 1) ^ 1) * 8 + (lane & 7));
        unsigned* const tkp = ctl(me, L_TK + d);
        unsigned call = 0, par = 0, ent = 0, d2 = 0, f2 = 0, one = 1;
        int ws = 0;
        auto fetch = [&](int u, bool ld) -> bool {
            const unsigned l = rfl(lb + (ws & 3) * 4096);
            if (u >= nin) {
                int qi = ((u - nin) >> 3) & 63;
                const int sub = (u - nin) & 7;
                const bool side = (d & 6) == 2;
                if (side) qi = ((relay_seq() >> (4 * ((qi >> 1) & 7))) & 7) * 2 + (qi & 1);
                if (side && qi >= 8) {
                    const int sl = qi & 7, t = (2 * (sl & 1) + ((d & 1) ^ 1)) * 16 + RC0 + (sl >> 1);
                    if ((unsigned)__builtin_amdgcn_readlane(ent, 48 + sl) < 8 * call) return false;
                    d2 = OFF_PS + t * 32768u + sub * 4096u; f2 = L_FP + t;
                    glds4k_s(me + par + OFF_RI + (((d & 1) ^ 1) * 8 + sl) * 32768u + sub * 4096u, vl, l);
                } else {
                    const unsigned e = __builtin_amdgcn_readlane(ent, qi);
                    if (e < call << 8) return false;
                    const int t = (e & 255) < 192 ? (e & 255) : 0, pi = ps_index(rank, t), ri = (d & 1) * 8 + (qi & 7);
                    d2 = (side ? OFF_RI + ri * 32768u : OFF_PS + pi * 32768u) + sub * 4096u; f2 = side ? L_RF + ri : L_FP + pi;
                    glds4k_s(me + OFF_LP + t * 32768u + sub * 4096u, vl, l);
                }
            } else {
                int q = u >> 4; q += q >= sk1; q += q >= sk2;
                if (q >= 28) q = 28 + ((q + ROT_O * d) & 3);
                else if (q >= 28 - ROT_N) { q += rot; q -= q >= 28 ? ROT_N : 0; }
                const int bk = order_at(q & 31), blk = bk & 15, sub = u & 15;
                if (bk >= 16) {
                    if (ld && (unsigned)__builtin_amdgcn_readlane(ent, 32 + blk) < 16 * call) return false;
                    d2 = OFF_WS + d * MiB + blk * 65536u + sub * 4096u; f2 = L_CW + d * 16 + blk;
                    if (ld) glds4k_s(me + par + OFF_WS + lseg * MiB + blk * 65536u + sub * 4096u, vl, l);
                } else {
                    d2 = OFF_XS + d * MiB + blk * 65536u + sub * 4096u; f2 = L_CX + d * 16 + blk;
                    if (ld) glds4k_x((const char*)P.x + (brick_row(dst, blk) + 32 * (sub & 3)) * 1024 + (dst & 1) * 512 + (sub >> 2) * 128, vx, l);
                }
            }
            ++ws;
            return true;
        };
        bool v0 = false, v1 = false;
        unsigned d0 = 0, f0 = 0, d1 = 0, f1 = 0, tq = 0;
        int tk = u0, tl = 0;
        auto claim = [&]() {
            asm volatile("s_mov_b32 exec_lo, %3\n\ts_mov_b32 exec_hi, 0\n\tglobal_atomic_add %0, %1, %2, off sc0 sc1\n\ts_mov_b64 exec, -1"
                         : "+v"(tq) : "v"(tkp), "v"(one), "s"(rfl(1 + tl)) : "memory");
            tl ^= 1;
        };
        const int ns = d == 1 ? 2 : 6;
        int sp = 0, rs = 0;
        unsigned pf = ~0u, cf = ~0u, idle = 0, tnext = 0;
        auto send = [&]() {
            unsigned now = (unsigned)__builtin_amdgcn_s_memrealtime();
            while ((int)(now - tnext) < 0) { __builtin_amdgcn_s_sleep(1); now = (unsigned)__builtin_amdgcn_s_memrealtime(); }
            tnext = ((int)(now - tnext) > PACE_SLACK ? now - PACE_SLACK : tnext) + PACE;
#pragma unroll
            for (int q = 0; q < 4; ++q) __builtin_amdgcn_raw_buffer_store_b128(o[q], rd, vl + par + d0 + q * 1024, 0, AUX_SYS);
        };
        snap[lane] = 0; snap[64 + lane] = 0;
        if (d > 1) {
            unsigned raw[16], raw2[16];
            const Piece p1 = {1, d, ORDER[1] & 15, u0 & 15}, p3 = {1, d, ORDER[3] & 15, u0 & 15};
            piece_load(P, p1, dst, lane, raw);
            piece_load(P, p3, dst, lane, raw2);
            fetch(u0, false);
            d0 = d2; cf = f2;
            fetch(u0 + 32, false);
            v1 = true; d1 = d2; f1 = f2;
            __builtin_amdgcn_sched_barrier(0);
            call = rfl(seq0) + 1; par = (call & 1) * PAR;
            WAIT_V(36);
            tnext = (unsigned)__builtin_amdgcn_s_memrealtime() + ((u0 & 7) * 2 + (u0 >> 3)) * PACE / 16 - STAG0;
#pragma unroll
            for (int q = 0; q < 4; ++q) o[q] = *(const u32x4*)(lds + wave * 32768 + q * 1024 + vl);
            send();
            __builtin_amdgcn_sched_barrier(0);
            fetch(u0 + 16, false);
            v0 = true; d0 = d2; f0 = f2; ws = 3; rs = 1; tk = u0 + 48;
            piece_xform(p1, raw, o);
#pragma unroll
            for (int q = 0; q < 4; ++q) *(u32x4*)(lds + wave * 32768 + 4096 + q * 1024 + vl) = o[q];
            piece_xform(p3, raw2, o);
#pragma unroll
            for (int q = 0; q < 4; ++q) *(u32x4*)(lds + wave * 32768 + 3 * 4096 + q * 1024 + vl) = o[q];
        } else {
            call = rfl(seq0) + 1; par = (call & 1) * PAR;
            asm volatile("" : "+s"(call), "+s"(par));
        }
        const unsigned tofs = ns * nw - (call - 1) * (nend - (ns - 2) * nw);
        const unsigned long long t0 = __builtin_amdgcn_s_memrealtime();
        if (d <= 1) tnext = (unsigned)t0;
        while (nend) {
            const bool S = v0, F = pf != ~0u;
            const unsigned sv = snap[sp * 64 + lane];
            if (S) {
#pragma unroll
                for (int q = 0; q < 4; ++q) o[q] = *(const u32x4*)(lds + wave * 32768 + (rs & 3) * 4096 + q * 1024 + vl);
                ++rs;
            }
            if (F) asm volatile("s_mov_b64 exec, 1\n\tglobal_atomic_add %0, %1, off sc1\n\ts_mov_b64 exec, -1" :: "v"(ctl(P.ws[dst & 7], pf)), "v"(one) : "memory");
            if (!S && !v1 && cf == ~0u && tk >= nend) break;
            ent = sv > ent ? sv : ent;
            if (S) send();
            const bool H = f0 - L_FP < 192u, E = S && H && ((d0 >> 12) & 7) == HINT;
            if (E) asm volatile("s_mov_b64 exec, 1\n\tglobal_atomic_add %0, %1, off sc1\n\ts_mov_b64 exec, -1" :: "v"(ctl(P.ws[dst & 7], f0)), "v"(one) : "memory");
            asm volatile("" : "+v"(ent) :: "memory");
            const bool ld = ws >= ns - 2, L = tk < nend && fetch(tk, ld);
            if (L && ld) claim();
            glds_word(rp, rfl(lb + 16384 + sp * 256));
            if (!S) WAIT_V(0);
            else if (L && ld) { if (F && E) WAIT_V(12); else if (F || E) WAIT_V(11); else WAIT_V(10); }
            else { if (F && E) WAIT_V(7); else if (F || E) WAIT_V(6); else WAIT_V(5); }
            asm volatile("" : "+v"(tq) :: "memory");
            pf = cf; cf = S && !H ? f0 : ~0u;
            v0 = v1; d0 = d1; f0 = f1;
            v1 = L; d1 = d2; f1 = f2;
            if (S) sp ^= 1;
            else if (!v0) { v0 = v1; d0 = d1; f0 = f1; v1 = false; }
            if (L) tk = ws < ns ? u0 + ws * nw : (int)(__builtin_amdgcn_readlane(tq, rfl(tl)) + tofs);
            else if (!S && (++idle & 63) == 0) {
                const unsigned t = rfl(__hip_atomic_load(tmo, __ATOMIC_RELAXED, __HIP_MEMORY_SCOPE_AGENT));
                if (t == call || __builtin_amdgcn_s_memrealtime() - t0 > DEADLINE) {
                    __hip_atomic_store(tmo, call, __ATOMIC_RELAXED, __HIP_MEMORY_SCOPE_AGENT);
                    break;
                }
            }
        }
        TSW(3);
        if (threadIdx.x == 0) __hip_atomic_store(ctl(me, L_SEQ + b), call, __ATOMIC_RELAXED, __HIP_MEMORY_SCOPE_AGENT);
        return;
    }

    const int grp = (wi - 8) >> 2, own = grp >= 4 ? 0 : 1 + (grp & 1);
    const int r4 = 2 * (xcd & 1) + (wi & 1), cb = 8 * (grp & 1) + 2 * (wi & 2) + (xcd >> 1);
    const int ro = r4, co = 4 * (grp & 2) + 2 * (wi & 2) + (xcd >> 1);
    for (int i = threadIdx.x; i < 512; i += 256) ((unsigned*)(lds + LDS_POLL))[i] = 0;
    bool dead = false;
    unsigned call, par;
    {
        const int n = (b - 64) * 4 + wave;
        Piece p0 = decode_local(n), p1 = decode_local(n + 768);
        auto stage = [&](int kind) {
            unsigned raw[16], raw2[16];
            p0.kind = p1.kind = kind;
            piece_load(P, p0, rank, lane, raw);
            piece_load(P, p1, rank, lane, raw2);
            __builtin_amdgcn_sched_barrier(0);
            call = rfl(seq0) + 1; par = (call & 1) * PAR;
            piece_xform(p0, raw, o);
            piece_store(P, p0, lane, par, o);
            piece_xform(p1, raw2, o);
            piece_store(P, p1, lane, par, o);
        };
        if (n < 512) stage(1); else stage(0);
        WAIT_V(4);
        piece_flag(P, p0, lane);
        WAIT_V(0);
        piece_flag(P, p1, lane);
        TS(1);
    }

    f32x16 acc[2][2], acc2[2][2];
#pragma unroll
    for (int m = 0; m < 2; ++m)
#pragma unroll
        for (int i = 0; i < 2; ++i)
#pragma unroll
            for (int e = 0; e < 16; ++e) { acc[m][i][e] = 0.f; acc2[m][i][e] = 0.f; }

    const int step0 = own == 2 ? 0 : 1, step1 = own == 1 ? 3 : 2;
    int pslot = 0;
    for (int step = step0; step < step1; ++step) {
        unsigned rem = 0x5555; int wofs = 0, rb = 12 + ro;
        const int cc = step == 1 ? cb : co;
        if (step == 0) { rem = 5; wofs = 8; }
        else if (step == 1) {
            rb = 4 * (grp >> 1) + r4;
            if (grp >= 2) rem &= ~(0x100u << (2 * r4));
        }
        const unsigned* cptr = ctl(me, (lane & 1) ? L_CW + ((((lane >> 1) & 7) + wofs) % 10) * 16 + cc : L_CX + ((lane >> 1) & 7) * 16 + rb);
        const unsigned target = 16 * call;
        const char* gA = me + par + OFF_XS + rb * 65536u + wave * 4096 + lane * 16;
        const char* gB = me + par + OFF_WS + wofs * MiB + cc * 65536u + wave * 4096 + lane * 16;
        const unsigned lw = ldsb + wave * 4096;
        const char* la = lds + (wave >> 1) * 8192 + lane * 16;
        const char* lb = lds + 16384 + (wave & 1) * 8192 + lane * 16;
        unsigned known = 0;
        bool polled = false;
        auto next = [&]() -> int {
            if (polled) {
                unsigned v = ((const unsigned*)(lds + LDS_POLL + pslot * 64))[lane & 15];
                known |= (unsigned)__builtin_amdgcn_ballot_w64(v >= target) & 0xffffu;
                ++pslot; polled = false;
            }
            if (!(known & (known >> 1) & rem)) {
                unsigned* mw = (unsigned*)(lds + LDS_POLL + pslot * 64);
                if (wave == 0) {
                    unsigned ok = (unsigned)wave_wait(cptr, lane < 16, target, rem, tmo, call);
                    if (lane == 0) *mw = ok ? ok : 0x80000000u;
                }
                BAR();
                unsigned ok = rfl(*mw);
                if (ok & 0x80000000u) { dead = true; ok = 0xffffu; }
                known |= ok;
                ++pslot;
            }
            const int s2 = __builtin_ctz(known & (known >> 1) & rem);
            rem &= ~(1u << s2);
            return s2 >> 1;
        };
        bf16x8 fa[2][4], fb[2][4];
#define SB() __builtin_amdgcn_sched_barrier(0)
#define RD(h, slot) do { _Pragma("unroll") for (int k2 = 0; k2 < 2; ++k2) { \
            fa[h][2 * k2] = *(const bf16x8*)(la + (slot) * STAGE_B + (2 * h + k2) * 1024); \
            fb[h][2 * k2] = *(const bf16x8*)(lb + (slot) * STAGE_B + h * 4096 + k2 * 1024); \
            fb[h][2 * k2 + 1] = *(const bf16x8*)(lb + (slot) * STAGE_B + h * 4096 + 2048 + k2 * 1024); \
            fa[h][2 * k2 + 1] = *(const bf16x8*)(la + (slot) * STAGE_B + 4096 + (2 * h + k2) * 1024); } SB(); } while (0)
#define MM(h, k2, m) do { acc[m][0] = __builtin_amdgcn_mfma_f32_32x32x16_bf16(fa[h][2 * k2 + m], fb[h][2 * k2], acc[m][0], 0, 0, 0); \
            acc[m][1] = __builtin_amdgcn_mfma_f32_32x32x16_bf16(fa[h][2 * k2 + m], fb[h][2 * k2 + 1], acc[m][1], 0, 0, 0); SB(); } while (0)
#define MMA(h) do { MM(h, 0, 0); MM(h, 0, 1); MM(h, 1, 0); MM(h, 1, 1); } while (0)
#define MMA_LD(h, g, l) do { const char* g_ = (g); const unsigned l_ = (l); \
            MM(h, 0, 0); glds1k<0>(g_, l_); SB(); MM(h, 0, 1); glds1k<1024>(g_, l_); SB(); \
            MM(h, 1, 0); glds1k<2048>(g_, l_); SB(); MM(h, 1, 1); glds1k<3072>(g_, l_); SB(); } while (0)
        BAR();
        unsigned so = next() * MiB;
#pragma unroll
        for (int kc = 0; kc < 3; ++kc) { glds4k(gA + so + kc * 16384, lw + kc * STAGE_B); glds4k(gB + so + kc * 16384, lw + kc * STAGE_B + 16384); }
        glds4k(gA + so + 3 * 16384, lw + 3 * STAGE_B);
        WAIT_V(20); BAR();
        RD(0, 0);
        while (rem) {
            RD(1, 0); MMA_LD(0, gB + so + 3 * 16384, lw + 3 * STAGE_B + 16384);
            WAIT_V(16); BAR();
            RD(0, 1);
            const unsigned no = next() * MiB;
            SB(); MMA_LD(1, gA + no, lw);
            RD(1, 1); MMA_LD(0, gB + no, lw + 16384);
            WAIT_V(16); BAR();
            RD(0, 2); MMA_LD(1, gA + no + 16384, lw + STAGE_B);
            RD(1, 2);
            if (rem && !(known & (known >> 1) & rem)) {
                const unsigned pa = rfl(ldsb + LDS_POLL + pslot * 64);
                if (wave == 0 && lane < 16) glds_word(cptr, pa);
                polled = true;
            }
            SB();
            MMA_LD(0, gB + no + 16384, lw + STAGE_B + 16384);
            WAIT_V(16); BAR();
            RD(0, 3); MMA_LD(1, gA + no + 2 * 16384, lw + 2 * STAGE_B);
            RD(1, 3); MMA_LD(0, gB + no + 2 * 16384, lw + 2 * STAGE_B + 16384);
            WAIT_V(16); BAR();
            RD(0, 0); MMA_LD(1, gA + no + 3 * 16384, lw + 3 * STAGE_B);
            so = no;
        }
        RD(1, 0); MMA_LD(0, gB + so + 3 * 16384, lw + 3 * STAGE_B + 16384);
        WAIT_V(16); BAR();
        RD(0, 1); MMA(1);
        RD(1, 1); MMA(0);
        WAIT_V(8); BAR();
        RD(0, 2); MMA(1);
        RD(1, 2); MMA(0);
        WAIT_V(0); BAR();
        RD(0, 3); MMA(1);
        RD(1, 3); MMA(0);
        MMA(1);
        TS(4 + step);
        if (step == 1) {
            const bool relay = grp < 2 && (unsigned)(cb - RC0) < RCN;
            const int d = relay ? 2 + (r4 & 1) : (grp < 2 ? 1 : 4 + r4), tile = rb * 16 + cb;
            unsigned pos = (cb - RC0) * 2 + (r4 >> 1);
            if (dead) poison(acc);
            __amdgpu_buffer_rsrc_t rs = rsrc(me, OFF_CTL);
#pragma unroll
            for (int q = 0; q < 8; ++q) {
                u32x4 v;
#pragma unroll
                for (int j = 0; j < 4; ++j) v[j] = __builtin_elementwise_max(pack2(acc[q >> 2][0][4 * (q & 3) + j], acc[q >> 2][1][4 * (q & 3) + j]), 1u);
                __builtin_amdgcn_raw_buffer_store_b128(v, rs, OFF_LP + tile * 32768u + wave * 8192 + lane * 16 + q * 1024, 0, LP_AUX);
            }
            __builtin_amdgcn_sched_barrier(0);
            if (threadIdx.x == 0 && !relay)
                pos = __hip_atomic_fetch_add(ctl(me, L_QT + d), 1u, __ATOMIC_RELAXED, __HIP_MEMORY_SCOPE_AGENT) - (call - 1) * (grp < 2 ? 64 - 4 * RCN : 32);
            WAIT_V(0);
            BAR();
            if (threadIdx.x == 0)
                __hip_atomic_store(ctl(me, L_Q + 2 * d) + (pos & 63), (call << 8) | tile, __ATOMIC_RELAXED, __HIP_MEMORY_SCOPE_AGENT);
            TS(7);
        }
        if (step < 2) {
#pragma unroll
            for (int m = 0; m < 2; ++m)
#pragma unroll
                for (int i = 0; i < 2; ++i) {
                    const f32x16 t = acc2[m][i];
                    acc2[m][i] = acc[m][i];
                    acc[m][i] = t;
                }
        }
    }

    if (own) {
        __amdgpu_buffer_rsrc_t rs = rsrc(me, OFF_LP);
        const int far = own - 1, tile = ro * 16 + co;
        const unsigned long long t0 = __builtin_amdgcn_s_memrealtime();
        for (unsigned rem = far ? 5 : 1; rem && !dead;) {
            const unsigned ok = (unsigned)wave_wait(ctl(me, L_FP + (far + ((lane >> 1) & far)) * 64 + tile), lane < 4, call, rem, tmo, call);
            if (!ok) { dead = true; break; }
            const int s2 = __builtin_ctz(ok & rem);
            TS(8 + 5 * far);
            const unsigned ofs = par + OFF_PS + ((far + (s2 >> 1)) * 64 + tile) * 32768u + wave * 8192;
            u32x4 pv[8];
#pragma unroll
            for (int q = 0; q < 8; ++q) pv[q] = __builtin_amdgcn_raw_buffer_load_b128(rs, lane * 16 + q * 1024, ofs, AUX_SYS);
            unsigned mn = ~0u;
#pragma unroll
            for (int q = 0; q < 8; ++q) mn = __builtin_elementwise_min(__builtin_elementwise_min(mn, __builtin_elementwise_min(pv[q][0], pv[q][1])), __builtin_elementwise_min(pv[q][2], pv[q][3]));
            if (__builtin_amdgcn_ballot_w64(mn == 0)) {
                if (__builtin_amdgcn_s_memrealtime() - t0 > DEADLINE) { __hip_atomic_store(tmo, call, __ATOMIC_RELAXED, __HIP_MEMORY_SCOPE_AGENT); dead = true; }
                continue;
            }
#pragma unroll
            for (int q = 0; q < 8; ++q) {
                __builtin_amdgcn_raw_buffer_store_b128(u32x4{0, 0, 0, 0}, rs, ofs + lane * 16 + q * 1024, 0, AUX_SYS);
#pragma unroll
                for (int j = 0; j < 4; ++j) {
                    acc[q >> 2][0][4 * (q & 3) + j] += __builtin_bit_cast(float, pv[q][j] << 16);
                    acc[q >> 2][1][4 * (q & 3) + j] += __builtin_bit_cast(float, pv[q][j] & 0xffff0000u);
                }
            }
            rem &= ~(1u << s2);
        }
        if (dead) poison(acc);
        const __amdgpu_buffer_rsrc_t ors = rsrc(P.out, 512 * 4096 * 2);
        const unsigned oo = (128 * ro + 64 * (wave >> 1) + (lane >> 3)) * 8192 + 4096 * ((rank >> 2) ^ far) + 256 * co + 128 * (wave & 1) + (lane & 7) * 16;
        unsigned* const lt = (unsigned*)(lds + wave * 8192) + (lane >> 5) * 128 + (lane & 31);
#pragma unroll
        for (int q = 0; q < 8; ++q) {
#pragma unroll
            for (int j = 0; j < 4; j += 2) {
                const int e = 4 * (q & 3) + j;
                const f32x2 g0 = gelu2(f32x2{acc[q >> 2][0][e], acc[q >> 2][0][e + 1]}), g1 = gelu2(f32x2{acc[q >> 2][1][e], acc[q >> 2][1][e + 1]});
                lt[q * 256 + j * 32] = pack2(g0[0], g1[0]);
                lt[q * 256 + j * 32 + 32] = pack2(g0[1], g1[1]);
            }
            __builtin_amdgcn_fence(__ATOMIC_ACQ_REL, "wavefront");
            const u32x4 v = *(const u32x4*)(lds + wave * 8192 + q * 1024 + lane * 16);
            __builtin_amdgcn_raw_buffer_store_b128(v, ors, oo + q * 65536, 0, OUT_AUX);
        }
        TS(9);
    }
    if (threadIdx.x == 0) __hip_atomic_store(ctl(me, L_SEQ + b), call, __ATOMIC_RELAXED, __HIP_MEMORY_SCOPE_AGENT);
}

extern "C" void kernel_launch(int rank, int nranks, ncclComm_t comm,
                              hipStream_t stream,
                              void* const* d_in, const int* in_sizes, int n_in,
                              void* d_out, int out_size,
                              const int* in_dtypes, int out_dtype,
                              void* const* d_out_all,
                              void* const* d_ws_all, size_t ws_bytes) {
    if (nranks != NR || (unsigned)rank >= (unsigned)NR || n_in != 2 || in_sizes[0] != 4096 * 512 || in_sizes[1] != 512 * 4096 ||
        out_size != 512 * 4096 || ws_bytes < WS_NEED) {
        fprintf(stderr, "kernel_launch: unexpected problem shape\n");
        return;
    }
    Params P;
    P.x = (const uint16_t*)d_in[0]; P.w = (const uint16_t*)d_in[1]; P.out = (uint16_t*)d_out;
    for (int p = 0; p < NR; ++p) P.ws[p] = (char*)d_ws_all[p];
    P.rank = rank; P.pad = 0; P.me = P.ws[rank];
    fused_gemm_rs<<<256, 256, 0, stream>>>(P);
    HIP_CHECK(hipGetLastError());
}
```

```cpp
#include <hip/hip_runtime.h>
#include <rccl/rccl.h>
#include <cstdio>
#include <stdint.h>

#define HIP_CHECK(expr) do { hipError_t _e = (expr); if (_e != hipSuccess) { \
    fprintf(stderr, "HIP error %s at %s:%d\n", hipGetErrorString(_e), __FILE__, __LINE__); \
    return; } } while (0)

typedef short bf16x8 __attribute__((ext_vector_type(8)));
typedef float f32x16 __attribute__((ext_vector_type(16)));
typedef unsigned u32x4 __attribute__((ext_vector_type(4)));
typedef __bf16 bf16x2 __attribute__((ext_vector_type(2)));
typedef float f32x2 __attribute__((ext_vector_type(2)));
#define DI __device__ __forceinline__

constexpr int NR = 8;
constexpr unsigned MiB = 1u << 20;
constexpr unsigned OFF_XS = 0;
constexpr unsigned OFF_WS = 8 * MiB;
constexpr unsigned OFF_PS = 18 * MiB;
constexpr unsigned OFF_RI = 24 * MiB;
constexpr unsigned PAR = 25 * MiB;
constexpr unsigned OFF_LP = 2 * PAR;
constexpr unsigned OFF_CTL = OFF_LP + 8 * MiB;
constexpr unsigned L_CX = 0;
constexpr unsigned L_CW = L_CX + 128;
constexpr unsigned L_FP = L_CW + 160;
constexpr unsigned L_SEQ = L_FP + 192;
constexpr unsigned L_TMO = L_SEQ + 256;
constexpr unsigned L_QT = L_TMO + 1;
constexpr unsigned L_Q = L_QT + 8;
constexpr unsigned L_TK = L_Q + 16;
constexpr unsigned L_RF = L_TK + 8;
constexpr unsigned L_END = L_RF + 16;
constexpr unsigned OFF_DBG = OFF_CTL + 512 * 1024;
constexpr unsigned WS_NEED = OFF_CTL + MiB;
static_assert(L_END * 128 <= 512 * 1024, "ctl");
#define LINK_LIST 1, 2, 3, 4, 5, 6, 7, 1, 2, 3, 4, 5, 6, 7, 0, 0, 0, 0, 0, 0, 0, 0, 0, 0, 0, 0, 0, 0, 0, 0, 0, 0
constexpr unsigned char LINKS[32] = {LINK_LIST};
constexpr unsigned long long lm16(int h, int idx) {
    unsigned long long v = 0;
    for (int i = 0; i < 16; ++i) { int c = 0; for (int j = 0; j < 16 * h + i; ++j) c += LINKS[j] == LINKS[16 * h + i]; v |= (unsigned long long)(idx ? (c & 7) : LINKS[16 * h + i]) << (4 * i); }
    return v;
}
constexpr unsigned lmcnt() { unsigned v = 0; for (int i = 0; i < 32; ++i) if (LINKS[i]) v += 1u << (4 * LINKS[i]); return v; }
#define RC0 9
#define RLAG RCN
#define RCN 3
#define MAPV 2
#define OUT_AUX 16
#define PACE 112
#define PACE_SLACK 20
#define STAG0 60
constexpr unsigned relay_seq() { unsigned v = 0; int n = 0; for (int i = 0; i < RCN + RLAG; ++i) { if (i < RCN) v |= (unsigned)i << (4 * n++); if (i >= RLAG) v |= (4u + i - RLAG) << (4 * n++); } return v; }
#define HINT 2
#define POLL_SLEEP 1
#define ROT_N 5
#define ROT_O 1
#define LP_AUX 16
#define PACE1 PACE
#define PACES PACE
#define PACE2 PACE
#define RETRY 1
#define ST_AUX 17
constexpr unsigned long long DEADLINE = 100000000ull;
constexpr int AUX_SYS = 17;

constexpr int RING = 4, STAGE_B = 32768, LDS_POLL = RING * STAGE_B, LDS_BYTES = LDS_POLL + 2048 + 256;

struct Params {
    const uint16_t* x; const uint16_t* w; char* me;
    int rank, pad;
    uint16_t* out;
    char* ws[NR];
};

#define TS(i) do {} while (0)
#define TSW(i) do {} while (0)

DI unsigned* ctl(char* ws, unsigned line) { return (unsigned*)(ws + OFF_CTL + line * 128u); }
DI __amdgpu_buffer_rsrc_t rsrc(const void* p, unsigned bytes) { return __builtin_amdgcn_make_buffer_rsrc((void*)p, 0, bytes, 0x00020000); }
DI int rfl(int v) { return __builtin_amdgcn_readfirstlane(v); }
#define WAIT_V(n) asm volatile("s_waitcnt vmcnt(" #n ")" ::: "memory")
#define BAR() do { asm volatile("s_waitcnt lgkmcnt(0)" ::: "memory"); __builtin_amdgcn_s_barrier(); asm volatile("" ::: "memory"); } while (0)

#define LDFL " sc0 sc1"
DI void glds4k(const char* g, unsigned lds) {
    unsigned keep;
    asm volatile("s_mov_b32 %0, m0\n\ts_mov_b32 m0, %2\n\ts_nop 0\n\t"
                 "global_load_lds_dwordx4 %1, off" LDFL "\n\t"
                 "global_load_lds_dwordx4 %1, off offset:1024" LDFL "\n\t"
                 "global_load_lds_dwordx4 %1, off offset:2048" LDFL "\n\t"
                 "global_load_lds_dwordx4 %1, off offset:3072" LDFL "\n\t"
                 "s_mov_b32 m0, %0" : "=&s"(keep) : "v"(g), "s"(lds) : "memory");
}
template <int OFS> DI void glds1k(const char* g, unsigned lds) {
    unsigned keep;
    asm volatile("s_mov_b32 %0, m0\n\ts_mov_b32 m0, %2\n\ts_nop 0\n\t"
                 "global_load_lds_dwordx4 %1, off offset:%c3" LDFL "\n\t"
                 "s_mov_b32 m0, %0" : "=&s"(keep) : "v"(g), "s"(lds), "i"(OFS) : "memory");
}
DI void glds4k_s(const char* g, unsigned v, unsigned lds) {
    unsigned keep;
    asm volatile("s_mov_b32 %0, m0\n\ts_mov_b32 m0, %3\n\ts_nop 0\n\t"
                 "global_load_lds_dwordx4 %1, %2 sc0 sc1\n\t"
                 "global_load_lds_dwordx4 %1, %2 offset:1024 sc0 sc1\n\t"
                 "global_load_lds_dwordx4 %1, %2 offset:2048 sc0 sc1\n\t"
                 "global_load_lds_dwordx4 %1, %2 offset:3072 sc0 sc1\n\t"
                 "s_mov_b32 m0, %0" : "=&s"(keep) : "v"(v), "s"(g), "s"(lds) : "memory");
}
DI void glds4k_x(const char* g, unsigned v, unsigned lds) {
    unsigned keep;
    asm volatile("s_mov_b32 %0, m0\n\ts_mov_b32 m0, %3\n\ts_nop 0\n\t"
                 "global_load_lds_dwordx4 %1, %2\n\ts_add_u32 m0, m0, 992\n\ts_nop 0\n\t"
                 "global_load_lds_dwordx4 %1, %2 offset:32\n\ts_add_u32 m0, m0, 992\n\ts_nop 0\n\t"
                 "global_load_lds_dwordx4 %1, %2 offset:64\n\ts_add_u32 m0, m0, 992\n\ts_nop 0\n\t"
                 "global_load_lds_dwordx4 %1, %2 offset:96\n\t"
                 "s_mov_b32 m0, %0" : "=&s"(keep) : "v"(v), "s"(g), "s"(lds) : "memory", "scc");
}
DI void glds_word(const unsigned* g, unsigned lds) {
    unsigned keep;
    asm volatile("s_mov_b32 %0, m0\n\ts_mov_b32 m0, %2\n\ts_nop 0\n\t"
                 "global_load_lds_dword %1, off sc0 sc1\n\t"
                 "s_mov_b32 m0, %0" : "=&s"(keep) : "v"(g), "s"(lds) : "memory");
}

DI unsigned long long wave_wait(const unsigned* p, bool active, unsigned target, unsigned long long need, unsigned* tmo, unsigned call) {
    unsigned long long t0 = __builtin_amdgcn_s_memrealtime();
    for (unsigned it = 0;; ++it) {
        unsigned v = 0;
        if (active) v = __hip_atomic_load(p, __ATOMIC_RELAXED, __HIP_MEMORY_SCOPE_SYSTEM);
        unsigned long long ok = __builtin_amdgcn_ballot_w64(active && v >= target);
        if (ok & (ok >> 1) & need) return ok;
        if ((it & 63) == 63) {
            unsigned t = rfl(__hip_atomic_load(tmo, __ATOMIC_RELAXED, __HIP_MEMORY_SCOPE_AGENT));
            if (t == call || __builtin_amdgcn_s_memrealtime() - t0 > DEADLINE) {
                __hip_atomic_store(tmo, call, __ATOMIC_RELAXED, __HIP_MEMORY_SCOPE_AGENT);
                return 0;
            }
        }
        __builtin_amdgcn_s_sleep(POLL_SLEEP);
    }
}

DI int brick_row(int r, int rb) {
    if (rb < 4) return 512 * (r ^ 1) + 128 * rb;
    if (rb < 12) return 512 * (r ^ (4 + (rb & 3))) + 256 * ((r >> 1) & 1) + 128 * ((rb - 4) >> 2);
    return 512 * r + 128 * (rb - 12);
}

#define ORDER_LIST 0, 16, 1, 17, 2, 3, 18, 19, 4, 20, 5, 21, 6, 22, 7, 23, 8, 24, 9, 25, 10, 26, 11, 27, 28, 29, 30, 31, 12, 13, 14, 15
constexpr unsigned char ORDER[32] = {ORDER_LIST};
static_assert(ORDER[0] < 4 && ORDER[1] >= 16 && ORDER[2] < 4 && ORDER[3] >= 16 && (lmcnt() >> 8) == 0x222222, "fixed start of the operand links");
constexpr bool rot_ok() { for (int q = 28; q < 32; ++q) if (ORDER[q] < 12) return false; return true; }
static_assert(rot_ok(), "skipped blocks stay outside the last range");
constexpr unsigned long long ord8(int g) { unsigned long long v = 0; for (int i = 0; i < 8; ++i) v |= (unsigned long long)ORDER[8 * g + i] << (8 * i); return v; }
constexpr unsigned long long pos8() { unsigned long long v = 0; for (int q = 0; q < 32; ++q) if (ORDER[q] >= 4 && ORDER[q] < 12) v |= (unsigned long long)q << (8 * (ORDER[q] - 4)); return v; }
DI int order_at(int q) {
    const unsigned long long v = (q & 16) ? ((q & 8) ? ord8(3) : ord8(2)) : ((q & 8) ? ord8(1) : ord8(0));
    return (int)(v >> (8 * (q & 7))) & 31;
}
struct Piece { int kind; int seg; int blk; int sub; };
DI Piece decode_local(int n) {
    Piece p;
    int rnd = n >= 768; n -= rnd * 768;
    p.sub = n & 15;
    if (n < 512) { p.kind = 1; p.seg = (n >> 8) * 8 + ((n >> 7) & 1); p.blk = rnd * 8 + ((n >> 4) & 7); }
    else { n -= 512; p.kind = 0; p.seg = n >> 7; int q = (n >> 4) & 7; p.blk = rnd ? 4 + q : (q < 4 ? q : q + 8); }
    return p;
}
DI void piece_load(const Params& P, const Piece& p, int dst, int lane, unsigned (&r)[16]) {
    const int k0 = 256 * ((p.seg >= 2 && p.seg < 8) ? (dst & 1) : (p.seg & 1)) + 64 * (p.sub >> 2);
    if (p.kind == 0) {
        const int row = brick_row(dst, p.blk) + 32 * (p.sub & 3) + (lane & 31);
        const u32x4* s = (const u32x4*)(P.x + (size_t)row * 512 + k0 + 8 * (lane >> 5));
#pragma unroll
        for (int ks = 0; ks < 4; ++ks) { u32x4 v = s[2 * ks]; r[4 * ks] = v[0]; r[4 * ks + 1] = v[1]; r[4 * ks + 2] = v[2]; r[4 * ks + 3] = v[3]; }
    } else {
        const int col = 2048 * ((dst >> 2) ^ (p.seg >> 3)) + 128 * p.blk + 64 * ((p.sub >> 1) & 1) + 2 * (lane & 31);
        const int k = k0 + 32 * (p.sub & 1) + 8 * (lane >> 5);
        const unsigned* s = (const unsigned*)(P.w + (size_t)k * 4096 + col);
#pragma unroll
        for (int ks = 0; ks < 2; ++ks)
#pragma unroll
            for (int j = 0; j < 8; ++j) r[8 * ks + j] = s[(16 * ks + j) * 2048];
    }
}
DI int ps_index(int r, int t) {
    const int rb = t >> 4, cb = t & 15;
    return rb < 4 ? t : (1 + (r & 1)) * 64 + (2 * ((r >> 1) & 1) + ((rb - 4) >> 2)) * 16 + cb;
}
DI void piece_xform(const Piece& p, const unsigned (&r)[16], u32x4 (&o)[4]) {
    if (p.kind == 1) {
#pragma unroll
        for (int ks = 0; ks < 2; ++ks)
#pragma unroll
            for (int m = 0; m < 4; ++m) {
                o[ks][m] = __builtin_amdgcn_perm(r[8 * ks + 2 * m + 1], r[8 * ks + 2 * m], 0x05040100u);
                o[2 + ks][m] = __builtin_amdgcn_perm(r[8 * ks + 2 * m + 1], r[8 * ks + 2 * m], 0x07060302u);
            }
    } else {
#pragma unroll
        for (int ks = 0; ks < 4; ++ks) o[ks] = u32x4{r[4 * ks], r[4 * ks + 1], r[4 * ks + 2], r[4 * ks + 3]};
    }
    asm volatile("" : "+v"(o[0]), "+v"(o[1]), "+v"(o[2]), "+v"(o[3]) :: "memory");
}
DI void piece_store(const Params& P, const Piece& p, int lane, unsigned par, const u32x4 (&o)[4]) {
    __amdgpu_buffer_rsrc_t rs = rsrc(P.me, OFF_LP);
    const unsigned off = par + p.sub * 4096u + (p.kind ? OFF_WS : OFF_XS) + p.seg * MiB + p.blk * 65536u;
#pragma unroll
    for (int q = 0; q < 4; ++q) __builtin_amdgcn_raw_buffer_store_b128(o[q], rs, off + lane * 16 + q * 1024, 0, ST_AUX);
}
DI void piece_flag(const Params& P, const Piece& p, int lane) {
    const unsigned line = (p.kind ? L_CW : L_CX) + p.seg * 16 + p.blk;
    if (lane == 0) __hip_atomic_fetch_add(ctl(P.me, line), 1u, __ATOMIC_RELAXED, __HIP_MEMORY_SCOPE_SYSTEM);
}

DI unsigned pack2(float a, float b) { f32x2 v = {a, b}; return __builtin_bit_cast(unsigned, __builtin_convertvector(v, bf16x2)); }
DI f32x2 gelu2(f32x2 y) {
    const float c2 = -2.0f * 0.7978845608028654f * 1.4426950408889634f;
    const f32x2 t = (y * y * (0.044715f * c2) + c2) * y;
    const f32x2 d = f32x2{__builtin_amdgcn_exp2f(t[0]), __builtin_amdgcn_exp2f(t[1])} + 1.0f;
    return y * f32x2{__builtin_amdgcn_rcpf(d[0]), __builtin_amdgcn_rcpf(d[1])};
}

DI void poison(f32x16 (&a)[2][2]) {
#pragma unroll
    for (int m = 0; m < 2; ++m)
#pragma unroll
        for (int i = 0; i < 2; ++i)
#pragma unroll
            for (int e = 0; e < 16; ++e) a[m][i][e] = __builtin_nanf("");
}

__global__ __launch_bounds__(256, 1) void fused_gemm_rs(Params P) {
    __shared__ __attribute__((aligned(1024))) char lds[LDS_BYTES];
    const int b = blockIdx.x, lane = threadIdx.x & 63, wave = rfl(threadIdx.x >> 6);
    const int xcd = b & 7, wi = b >> 3;
    const int rank = P.rank;
    char* const me = P.me;
    TS(0);
    const unsigned seq0 = __hip_atomic_load(ctl(me, L_SEQ + b), __ATOMIC_RELAXED, __HIP_MEMORY_SCOPE_AGENT);
    unsigned* const tmo = ctl(me, L_TMO);
    u32x4 o[4];

    const unsigned ldsb = (unsigned)(size_t)lds;
    if (wi < 8) {
        const int i = wave * 8 + wi;
        const int d = (int)((i & 16 ? lm16(1, 0) : lm16(0, 0)) >> (4 * (i & 15))) & 7, dst = rank ^ d;
        const int u0 = (int)(((i & 16 ? lm16(1, 1) : lm16(0, 1)) >> (4 * (i & 15))) & 7) * 8 + xcd;
        const unsigned vx = (lane & 31) * 1024 + (lane >> 5) * 16, vl = lane * 16;
        const unsigned lb = ldsb + wave * 32768;
        if (d > 1) {
            const char* xs = (const char*)P.x + (512 * (dst ^ 1) + 32 * (u0 & 3)) * 1024 + (dst & 1) * 512 + ((u0 >> 2) & 3) * 128;
            glds4k_x(xs + ORDER[0] * 131072, vx, lb);
            glds4k_x(xs + ORDER[2] * 131072, vx, lb + 8192);
        }
        __builtin_amdgcn_sched_barrier(0);
        const int nw = 8 * ((lmcnt() >> (4 * d)) & 15);
        const int nin = d == 1 ? 0 : (d < 4 ? 512 : 480);
        const int nend = d == 0 ? 0 : nin + (d == 1 ? 512 - 32 * RCN : (d < 4 ? 32 * RCN : 256));
        int sk1 = 99, sk2 = 99;
        const int rot = (d < 2 ? 0 : 7 - d) * ROT_N / 6;
        auto unrot = [&](int a) { return a < 28 - ROT_N ? a : (a - rot < 28 - ROT_N ? a - rot + ROT_N : a - rot); };
        if (d >= 4) { const int a = unrot((pos8() >> (8 * (d - 4))) & 31), c = unrot((pos8() >> (8 * d)) & 31); sk1 = a < c ? a : c; sk2 = a < c ? c : a; }
        const int lseg = (d >> 2) * 8 + (dst & 1);
        const __amdgpu_buffer_rsrc_t rd = rsrc(P.ws[dst & 7], OFF_LP);
        unsigned* const snap = (unsigned*)(lds + wave * 32768 + 16384);
        const unsigned* const rp = (d == 1 || lane < 32) ? ctl(me, L_Q + 2 * d) + lane
                                 : ctl(me, lane < 48 ? L_CW + lseg * 16 + (lane & 15) : L_RF + ((d & 1) ^ 1) * 8 + (lane & 7));
        unsigned* const tkp = ctl(me, L_TK + d);
        unsigned call = 0, par = 0, ent = 0, d2 = 0, f2 = 0, one = 1;
        int ws = 0;
        auto fetch = [&](int u, bool ld) -> bool {
            const unsigned l = rfl(lb + (ws & 3) * 4096);
            if (u >= nin) {
                int qi = ((u - nin) >> 3) & 63;
                const int sub = (u - nin) & 7;
                const bool side = (d & 6) == 2;
                if (side) qi = ((relay_seq() >> (4 * ((qi >> 1) & 7))) & 7) * 2 + (qi & 1);
                if (side && qi >= 8) {
                    const int sl = qi & 7, t = (2 * (sl & 1) + ((d & 1) ^ 1)) * 16 + RC0 + (sl >> 1);
                    if ((unsigned)__builtin_amdgcn_readlane(ent, 48 + sl) < 8 * call) return false;
                    d2 = OFF_PS + t * 32768u + sub * 4096u; f2 = L_FP + t;
                    glds4k_s(me + par + OFF_RI + (((d & 1) ^ 1) * 8 + sl) * 32768u + sub * 4096u, vl, l);
                } else {
                    const unsigned e = __builtin_amdgcn_readlane(ent, qi);
                    if (e < call << 8) return false;
                    const int t = (e & 255) < 192 ? (e & 255) : 0, pi = ps_index(rank, t), ri = (d & 1) * 8 + (qi & 7);
                    d2 = (side ? OFF_RI + ri * 32768u : OFF_PS + pi * 32768u) + sub * 4096u; f2 = side ? L_RF + ri : L_FP + pi;
                    glds4k_s(me + OFF_LP + t * 32768u + sub * 4096u, vl, l);
                }
            } else {
                int q = u >> 4; q += q >= sk1; q += q >= sk2;
                if (q >= 28) q = 28 + ((q + ROT_O * d) & 3);
                else if (q >= 28 - ROT_N) { q += rot; q -= q >= 28 ? ROT_N : 0; }
                const int bk = order_at(q & 31), blk = bk & 15, sub = u & 15;
                if (bk >= 16) {
                    if (ld && (unsigned)__builtin_amdgcn_readlane(ent, 32 + blk) < 16 * call) return false;
                    d2 = OFF_WS + d * MiB + blk * 65536u + sub * 4096u; f2 = L_CW + d * 16 + blk;
                    if (ld) glds4k_s(me + par + OFF_WS + lseg * MiB + blk * 65536u + sub * 4096u, vl, l);
                } else {
                    d2 = OFF_XS + d * MiB + blk * 65536u + sub * 4096u; f2 = L_CX + d * 16 + blk;
                    if (ld) glds4k_x((const char*)P.x + (brick_row(dst, blk) + 32 * (sub & 3)) * 1024 + (dst & 1) * 512 + (sub >> 2) * 128, vx, l);
                }
            }
            ++ws;
            return true;
        };
        bool v0 = false, v1 = false;
        unsigned d0 = 0, f0 = 0, d1 = 0, f1 = 0, tq = 0;
        int tk = u0, tl = 0;
        auto claim = [&]() {
            asm volatile("s_mov_b32 exec_lo, %3\n\ts_mov_b32 exec_hi, 0\n\tglobal_atomic_add %0, %1, %2, off sc0 sc1\n\ts_mov_b64 exec, -1"
                         : "+v"(tq) : "v"(tkp), "v"(one), "s"(rfl(1 + tl)) : "memory");
            tl ^= 1;
        };
        const int ns = d == 1 ? 2 : 6;
        int sp = 0, rs = 0;
        unsigned pf = ~0u, cf = ~0u, idle = 0, tnext = 0;
        auto send = [&]() {
            unsigned now = (unsigned)__builtin_amdgcn_s_memrealtime();
            while ((int)(now - tnext) < 0) { __builtin_amdgcn_s_sleep(1); now = (unsigned)__builtin_amdgcn_s_memrealtime(); }
            tnext = ((int)(now - tnext) > PACE_SLACK ? now - PACE_SLACK : tnext) + (d == 1 ? PACE1 : (d < 4 ? PACES : (d0 >= OFF_PS ? PACE2 : PACE)));
#pragma unroll
            for (int q = 0; q < 4; ++q) __builtin_amdgcn_raw_buffer_store_b128(o[q], rd, vl + par + d0 + q * 1024, 0, AUX_SYS);
        };
        snap[lane] = 0; snap[64 + lane] = 0;
        if (d > 1) {
            unsigned raw[16], raw2[16];
            const Piece p1 = {1, d, ORDER[1] & 15, u0 & 15}, p3 = {1, d, ORDER[3] & 15, u0 & 15};
            piece_load(P, p1, dst, lane, raw);
            piece_load(P, p3, dst, lane, raw2);
            fetch(u0, false);
            d0 = d2; cf = f2;
            fetch(u0 + 32, false);
            v1 = true; d1 = d2; f1 = f2;
            __builtin_amdgcn_sched_barrier(0);
            call = rfl(seq0) + 1; par = (call & 1) * PAR;
            WAIT_V(36);
            tnext = (unsigned)__builtin_amdgcn_s_memrealtime() + ((u0 & 7) * 2 + (u0 >> 3)) * PACE / 16 - STAG0;
#pragma unroll
            for (int q = 0; q < 4; ++q) o[q] = *(const u32x4*)(lds + wave * 32768 + q * 1024 + vl);
            send();
            __builtin_amdgcn_sched_barrier(0);
            fetch(u0 + 16, false);
            v0 = true; d0 = d2; f0 = f2; ws = 3; rs = 1; tk = u0 + 48;
            piece_xform(p1, raw, o);
#pragma unroll
            for (int q = 0; q < 4; ++q) *(u32x4*)(lds + wave * 32768 + 4096 + q * 1024 + vl) = o[q];
            piece_xform(p3, raw2, o);
#pragma unroll
            for (int q = 0; q < 4; ++q) *(u32x4*)(lds + wave * 32768 + 3 * 4096 + q * 1024 + vl) = o[q];
        } else {
            call = rfl(seq0) + 1; par = (call & 1) * PAR;
            asm volatile("" : "+s"(call), "+s"(par));
        }
        const unsigned tofs = ns * nw - (call - 1) * (nend - (ns - 2) * nw);
        const unsigned long long t0 = __builtin_amdgcn_s_memrealtime();
        if (d <= 1) tnext = (unsigned)t0;
        while (nend) {
            const bool S = v0, F = pf != ~0u;
            const unsigned sv = snap[sp * 64 + lane];
            if (S) {
#pragma unroll
                for (int q = 0; q < 4; ++q) o[q] = *(const u32x4*)(lds + wave * 32768 + (rs & 3) * 4096 + q * 1024 + vl);
                ++rs;
            }
            if (F) asm volatile("s_mov_b64 exec, 1\n\tglobal_atomic_add %0, %1, off sc1\n\ts_mov_b64 exec, -1" :: "v"(ctl(P.ws[dst & 7], pf)), "v"(one) : "memory");
            if (!S && !v1 && cf == ~0u && tk >= nend) break;
            ent = sv > ent ? sv : ent;
            if (S) send();
            const bool H = f0 - L_FP < 192u, E = S && H && ((d0 >> 12) & 7) == HINT;
            if (E) asm volatile("s_mov_b64 exec, 1\n\tglobal_atomic_add %0, %1, off sc1\n\ts_mov_b64 exec, -1" :: "v"(ctl(P.ws[dst & 7], f0)), "v"(one) : "memory");
            asm volatile("" : "+v"(ent) :: "memory");
            const bool ld = ws >= ns - 2, L = tk < nend && fetch(tk, ld);
            if (L && ld) claim();
            glds_word(rp, rfl(lb + 16384 + sp * 256));
            if (!S) WAIT_V(0);
            else if (L && ld) { if (F && E) WAIT_V(12); else if (F || E) WAIT_V(11); else WAIT_V(10); }
            else { if (F && E) WAIT_V(7); else if (F || E) WAIT_V(6); else WAIT_V(5); }
            asm volatile("" : "+v"(tq) :: "memory");
            pf = cf; cf = S && !H ? f0 : ~0u;
            v0 = v1; d0 = d1; f0 = f1;
            v1 = L; d1 = d2; f1 = f2;
            if (S) sp ^= 1;
            else if (!v0) { v0 = v1; d0 = d1; f0 = f1; v1 = false; }
            if (L) tk = ws < ns ? u0 + ws * nw : (int)(__builtin_amdgcn_readlane(tq, rfl(tl)) + tofs);
            else if (!S && (++idle & 63) == 0) {
                const unsigned t = rfl(__hip_atomic_load(tmo, __ATOMIC_RELAXED, __HIP_MEMORY_SCOPE_AGENT));
                if (t == call || __builtin_amdgcn_s_memrealtime() - t0 > DEADLINE) {
                    __hip_atomic_store(tmo, call, __ATOMIC_RELAXED, __HIP_MEMORY_SCOPE_AGENT);
                    break;
                }
            }
        }
        TSW(3);
        if (threadIdx.x == 0) __hip_atomic_store(ctl(me, L_SEQ + b), call, __ATOMIC_RELAXED, __HIP_MEMORY_SCOPE_AGENT);
        return;
    }

    const int grp = (wi - 8) >> 2, own = grp >= 4 ? 0 : 1 + (grp & 1);
    const int r4 = 2 * (xcd & 1) + (wi & 1), cb = 8 * (grp & 1) + 2 * (wi & 2) + (xcd >> 1);
    const int ro = r4, co = 4 * (grp & 2) + 2 * (wi & 2) + (xcd >> 1);
    for (int i = threadIdx.x; i < 512; i += 256) ((unsigned*)(lds + LDS_POLL))[i] = 0;
    bool dead = false;
    unsigned call, par;
    {
        const int n = (b - 64) * 4 + wave;
        Piece p0 = decode_local(n), p1 = decode_local(n + 768);
        auto stage = [&](int kind) {
            unsigned raw[16], raw2[16];
            p0.kind = p1.kind = kind;
            piece_load(P, p0, rank, lane, raw);
            piece_load(P, p1, rank, lane, raw2);
            __builtin_amdgcn_sched_barrier(0);
            call = rfl(seq0) + 1; par = (call & 1) * PAR;
            piece_xform(p0, raw, o);
            piece_store(P, p0, lane, par, o);
            piece_xform(p1, raw2, o);
            piece_store(P, p1, lane, par, o);
        };
        if (n < 512) stage(1); else stage(0);
        WAIT_V(4);
        piece_flag(P, p0, lane);
        WAIT_V(0);
        piece_flag(P, p1, lane);
        TS(1);
    }

    f32x16 acc[2][2], acc2[2][2];
#pragma unroll
    for (int m = 0; m < 2; ++m)
#pragma unroll
        for (int i = 0; i < 2; ++i)
#pragma unroll
            for (int e = 0; e < 16; ++e) { acc[m][i][e] = 0.f; acc2[m][i][e] = 0.f; }

    const int step0 = own == 2 ? 0 : 1, step1 = own == 1 ? 3 : 2;
    int pslot = 0;
    for (int step = step0; step < step1; ++step) {
        unsigned rem = 0x5555; int wofs = 0, rb = 12 + ro;
        const int cc = step == 1 ? cb : co;
        if (step == 0) { rem = 5; wofs = 8; }
        else if (step == 1) {
            rb = 4 * (grp >> 1) + r4;
            if (grp >= 2) rem &= ~(0x100u << (2 * r4));
        }
        const unsigned* cptr = ctl(me, (lane & 1) ? L_CW + ((((lane >> 1) & 7) + wofs) % 10) * 16 + cc : L_CX + ((lane >> 1) & 7) * 16 + rb);
        const unsigned target = 16 * call;
        const char* gA = me + par + OFF_XS + rb * 65536u + wave * 4096 + lane * 16;
        const char* gB = me + par + OFF_WS + wofs * MiB + cc * 65536u + wave * 4096 + lane * 16;
        const unsigned lw = ldsb + wave * 4096;
        const char* la = lds + (wave >> 1) * 8192 + lane * 16;
        const char* lb = lds + 16384 + (wave & 1) * 8192 + lane * 16;
        unsigned known = 0;
        bool polled = false;
        auto next = [&]() -> int {
            if (polled) {
                unsigned v = ((const unsigned*)(lds + LDS_POLL + pslot * 64))[lane & 15];
                known |= (unsigned)__builtin_amdgcn_ballot_w64(v >= target) & 0xffffu;
                ++pslot; polled = false;
            }
            if (!(known & (known >> 1) & rem)) {
                unsigned* mw = (unsigned*)(lds + LDS_POLL + pslot * 64);
                if (wave == 0) {
                    unsigned ok = (unsigned)wave_wait(cptr, lane < 16, target, rem, tmo, call);
                    if (lane == 0) *mw = ok ? ok : 0x80000000u;
                }
                BAR();
                unsigned ok = rfl(*mw);
                if (ok & 0x80000000u) { dead = true; ok = 0xffffu; }
                known |= ok;
                ++pslot;
            }
            const int s2 = __builtin_ctz(known & (known >> 1) & rem);
            rem &= ~(1u << s2);
            return s2 >> 1;
        };
        bf16x8 fa[2][4], fb[2][4];
#define SB() __builtin_amdgcn_sched_barrier(0)
#define RD(h, slot) do { _Pragma("unroll") for (int k2 = 0; k2 < 2; ++k2) { \
            fa[h][2 * k2] = *(const bf16x8*)(la + (slot) * STAGE_B + (2 * h + k2) * 1024); \
            fb[h][2 * k2] = *(const bf16x8*)(lb + (slot) * STAGE_B + h * 4096 + k2 * 1024); \
            fb[h][2 * k2 + 1] = *(const bf16x8*)(lb + (slot) * STAGE_B + h * 4096 + 2048 + k2 * 1024); \
            fa[h][2 * k2 + 1] = *(const bf16x8*)(la + (slot) * STAGE_B + 4096 + (2 * h + k2) * 1024); } SB(); } while (0)
#define MM(h, k2, m) do { acc[m][0] = __builtin_amdgcn_mfma_f32_32x32x16_bf16(fa[h][2 * k2 + m], fb[h][2 * k2], acc[m][0], 0, 0, 0); \
            acc[m][1] = __builtin_amdgcn_mfma_f32_32x32x16_bf16(fa[h][2 * k2 + m], fb[h][2 * k2 + 1], acc[m][1], 0, 0, 0); SB(); } while (0)
#define MMA(h) do { MM(h, 0, 0); MM(h, 0, 1); MM(h, 1, 0); MM(h, 1, 1); } while (0)
#define MMA_LD(h, g, l) do { const char* g_ = (g); const unsigned l_ = (l); \
            MM(h, 0, 0); glds1k<0>(g_, l_); SB(); MM(h, 0, 1); glds1k<1024>(g_, l_); SB(); \
            MM(h, 1, 0); glds1k<2048>(g_, l_); SB(); MM(h, 1, 1); glds1k<3072>(g_, l_); SB(); } while (0)
        BAR();
        unsigned so = next() * MiB;
#pragma unroll
        for (int kc = 0; kc < 3; ++kc) { glds4k(gA + so + kc * 16384, lw + kc * STAGE_B); glds4k(gB + so + kc * 16384, lw + kc * STAGE_B + 16384); }
        glds4k(gA + so + 3 * 16384, lw + 3 * STAGE_B);
        WAIT_V(20); BAR();
        RD(0, 0);
        while (rem) {
            RD(1, 0); MMA_LD(0, gB + so + 3 * 16384, lw + 3 * STAGE_B + 16384);
            WAIT_V(16); BAR();
            RD(0, 1);
            const unsigned no = next() * MiB;
            SB(); MMA_LD(1, gA + no, lw);
            RD(1, 1); MMA_LD(0, gB + no, lw + 16384);
            WAIT_V(16); BAR();
            RD(0, 2); MMA_LD(1, gA + no + 16384, lw + STAGE_B);
            RD(1, 2);
            if (rem && !(known & (known >> 1) & rem)) {
                const unsigned pa = rfl(ldsb + LDS_POLL + pslot * 64);
                if (wave == 0 && lane < 16) glds_word(cptr, pa);
                polled = true;
            }
            SB();
            MMA_LD(0, gB + no + 16384, lw + STAGE_B + 16384);
            WAIT_V(16); BAR();
            RD(0, 3); MMA_LD(1, gA + no + 2 * 16384, lw + 2 * STAGE_B);
            RD(1, 3); MMA_LD(0, gB + no + 2 * 16384, lw + 2 * STAGE_B + 16384);
            WAIT_V(16); BAR();
            RD(0, 0); MMA_LD(1, gA + no + 3 * 16384, lw + 3 * STAGE_B);
            so = no;
        }
        RD(1, 0); MMA_LD(0, gB + so + 3 * 16384, lw + 3 * STAGE_B + 16384);
        WAIT_V(16); BAR();
        RD(0, 1); MMA(1);
        RD(1, 1); MMA(0);
        WAIT_V(8); BAR();
        RD(0, 2); MMA(1);
        RD(1, 2); MMA(0);
        WAIT_V(0); BAR();
        RD(0, 3); MMA(1);
        RD(1, 3); MMA(0);
        MMA(1);
        TS(4 + step);
        if (step == 1) {
            const bool relay = grp < 2 && (unsigned)(cb - RC0) < RCN;
            const int d = relay ? 2 + (r4 & 1) : (grp < 2 ? 1 : 4 + r4), tile = rb * 16 + cb;
            unsigned pos = (cb - RC0) * 2 + (r4 >> 1);
            if (dead) poison(acc);
            __amdgpu_buffer_rsrc_t rs = rsrc(me, OFF_CTL);
#pragma unroll
            for (int q = 0; q < 8; ++q) {
                u32x4 v;
#pragma unroll
                for (int j = 0; j < 4; ++j) v[j] = __builtin_elementwise_max(pack2(acc[q >> 2][0][4 * (q & 3) + j], acc[q >> 2][1][4 * (q & 3) + j]), 1u);
                __builtin_amdgcn_raw_buffer_store_b128(v, rs, OFF_LP + tile * 32768u + wave * 8192 + lane * 16 + q * 1024, 0, LP_AUX);
            }
            __builtin_amdgcn_sched_barrier(0);
            if (threadIdx.x == 0 && !relay)
                pos = __hip_atomic_fetch_add(ctl(me, L_QT + d), 1u, __ATOMIC_RELAXED, __HIP_MEMORY_SCOPE_AGENT) - (call - 1) * (grp < 2 ? 64 - 4 * RCN : 32);
            WAIT_V(0);
            BAR();
            if (threadIdx.x == 0)
                __hip_atomic_store(ctl(me, L_Q + 2 * d) + (pos & 63), (call << 8) | tile, __ATOMIC_RELAXED, __HIP_MEMORY_SCOPE_AGENT);
            TS(7);
        }
        if (step < 2) {
#pragma unroll
            for (int m = 0; m < 2; ++m)
#pragma unroll
                for (int i = 0; i < 2; ++i) {
                    const f32x16 t = acc2[m][i];
                    acc2[m][i] = acc[m][i];
                    acc[m][i] = t;
                }
        }
    }

    if (own) {
        __amdgpu_buffer_rsrc_t rs = rsrc(me, OFF_LP);
        const int far = own - 1, tile = ro * 16 + co;
        const unsigned long long t0 = __builtin_amdgcn_s_memrealtime();
        unsigned ok = 0;
        for (unsigned rem = far ? 5 : 1; rem && !dead;) {
            if (!RETRY || !(ok & rem)) ok = (unsigned)wave_wait(ctl(me, L_FP + (far + ((lane >> 1) & far)) * 64 + tile), lane < 4, call, rem, tmo, call);
            if (!ok) { dead = true; break; }
            const int s2 = __builtin_ctz(ok & rem);
            TS(8 + 5 * far);
            const unsigned ofs = par + OFF_PS + ((far + (s2 >> 1)) * 64 + tile) * 32768u + wave * 8192;
            u32x4 pv[8];
#pragma unroll
            for (int q = 0; q < 8; ++q) pv[q] = __builtin_amdgcn_raw_buffer_load_b128(rs, lane * 16 + q * 1024, ofs, AUX_SYS);
            unsigned mn = ~0u;
#pragma unroll
            for (int q = 0; q < 8; ++q) mn = __builtin_elementwise_min(__builtin_elementwise_min(mn, __builtin_elementwise_min(pv[q][0], pv[q][1])), __builtin_elementwise_min(pv[q][2], pv[q][3]));
            if (__builtin_amdgcn_ballot_w64(mn == 0)) {
                if (__builtin_amdgcn_s_memrealtime() - t0 > DEADLINE) { __hip_atomic_store(tmo, call, __ATOMIC_RELAXED, __HIP_MEMORY_SCOPE_AGENT); dead = true; }
                continue;
            }
#pragma unroll
            for (int q = 0; q < 8; ++q) {
                __builtin_amdgcn_raw_buffer_store_b128(u32x4{0, 0, 0, 0}, rs, ofs + lane * 16 + q * 1024, 0, AUX_SYS);
#pragma unroll
                for (int j = 0; j < 4; ++j) {
                    acc[q >> 2][0][4 * (q & 3) + j] += __builtin_bit_cast(float, pv[q][j] << 16);
                    acc[q >> 2][1][4 * (q & 3) + j] += __builtin_bit_cast(float, pv[q][j] & 0xffff0000u);
                }
            }
            rem &= ~(1u << s2);
        }
        if (dead) poison(acc);
        const __amdgpu_buffer_rsrc_t ors = rsrc(P.out, 512 * 4096 * 2);
        const unsigned oo = (128 * ro + 64 * (wave >> 1) + (lane >> 3)) * 8192 + 4096 * ((rank >> 2) ^ far) + 256 * co + 128 * (wave & 1) + (lane & 7) * 16;
        unsigned* const lt = (unsigned*)(lds + wave * 8192) + (lane >> 5) * 128 + (lane & 31);
#pragma unroll
        for (int q = 0; q < 8; ++q) {
#pragma unroll
            for (int j = 0; j < 4; j += 2) {
                const int e = 4 * (q & 3) + j;
                const f32x2 g0 = gelu2(f32x2{acc[q >> 2][0][e], acc[q >> 2][0][e + 1]}), g1 = gelu2(f32x2{acc[q >> 2][1][e], acc[q >> 2][1][e + 1]});
                lt[q * 256 + j * 32] = pack2(g0[0], g1[0]);
                lt[q * 256 + j * 32 + 32] = pack2(g0[1], g1[1]);
            }
            __builtin_amdgcn_fence(__ATOMIC_ACQ_REL, "wavefront");
            const u32x4 v = *(const u32x4*)(lds + wave * 8192 + q * 1024 + lane * 16);
            __builtin_amdgcn_raw_buffer_store_b128(v, ors, oo + q * 65536, 0, OUT_AUX);
        }
        TS(9);
    }
    if (threadIdx.x == 0) __hip_atomic_store(ctl(me, L_SEQ + b), call, __ATOMIC_RELAXED, __HIP_MEMORY_SCOPE_AGENT);
}

extern "C" void kernel_launch(int rank, int nranks, ncclComm_t comm,
                              hipStream_t stream,
                              void* const* d_in, const int* in_sizes, int n_in,
                              void* d_out, int out_size,
                              const int* in_dtypes, int out_dtype,
                              void* const* d_out_all,
                              void* const* d_ws_all, size_t ws_bytes) {
    if (nranks != NR || (unsigned)rank >= (unsigned)NR || n_in != 2 || in_sizes[0] != 4096 * 512 || in_sizes[1] != 512 * 4096 ||
        out_size != 512 * 4096 || ws_bytes < WS_NEED) {
        fprintf(stderr, "kernel_launch: unexpected problem shape\n");
        return;
    }
    Params P;
    P.x = (const uint16_t*)d_in[0]; P.w = (const uint16_t*)d_in[1]; P.out = (uint16_t*)d_out;
    for (int p = 0; p < NR; ++p) P.ws[p] = (char*)d_ws_all[p];
    P.rank = rank; P.pad = 0; P.me = P.ws[rank];
    fused_gemm_rs<<<256, 256, 0, stream>>>(P);
    HIP_CHECK(hipGetLastError());
}
```
